```python
import numpy as np

kernel_name = "gpu_gemm_ar_m2048_k2048_n2048_f32_gelu"


ABSMAX_RTOL = 0.001

COMPUTE_DTYPE = "float32"
EPILOGUE = "gelu"
IN_QUANT = None
OUT_QUANT = None
IN_DTYPES = {"x": "float32", "w_mat": "float32"}
WORLD_SIZE = 8

def setup_inputs():

    x = np.random.randn(2048, 2048).astype(np.float32)
    w_mat = (np.random.randn(2048, 2048) / np.sqrt(2048)).astype(np.float32)
    return {"x": x, "w_mat": w_mat}

def _epilogue(y):
    c = 0.7978845608028654
    return (0.5 * y * (1.0 + np.tanh(c * (y + 0.044715 * y ** 3)))).astype(np.float32)

def reference(x, w_mat):
    y = (x @ w_mat).astype(np.float32)
    return _epilogue(y)
```

```cpp
#include <hip/hip_runtime.h>
#include <rccl/rccl.h>
#include <cstdio>
#include <cstdint>

#define HIP_CHECK(expr) do { hipError_t _e = (expr); if (_e != hipSuccess) { \
    fprintf(stderr, "HIP error %s at %s:%d\n", hipGetErrorString(_e), __FILE__, __LINE__); \
    return; } } while (0)

typedef _Float16 h8 __attribute__((ext_vector_type(8)));
typedef _Float16 h2 __attribute__((ext_vector_type(2)));
typedef float f4 __attribute__((ext_vector_type(4)));
typedef float f2 __attribute__((ext_vector_type(2)));
typedef unsigned u4 __attribute__((ext_vector_type(4)));
typedef unsigned u2 __attribute__((ext_vector_type(2)));

namespace {

constexpr int NR = 8;
constexpr int MM = 2048, NN = 2048, KS = 256;
constexpr int NWG = 256;
constexpr int NWAVE = 14;
constexpr int NSTREAM = 16;
constexpr int NPUSHWG = NSTREAM * NR;
constexpr int NPIECE = 512;
constexpr int PPS = NPIECE / NSTREAM;

constexpr unsigned CTR_OFF  = 0;
constexpr unsigned TMO_OFF  = 32768;
constexpr unsigned FLAG_OFF = 65536;
constexpr unsigned DBG_OFF  = 262144;
constexpr unsigned DATA_OFF = 1u << 20;
constexpr unsigned PAR_BYTES = 16u << 20;
constexpr unsigned TYPE_BYTES = 8u << 20;
constexpr unsigned SRC_BYTES = 1u << 20;
constexpr unsigned WS_NEED = DATA_OFF + 2 * PAR_BYTES;

constexpr int AUX_SYS = 17;
constexpr unsigned RSRC3 = 0x00020000u;
constexpr unsigned long long DEADLINE = 100000000ull;
constexpr unsigned CALL_MASK = 0x3FFFFFu;
constexpr unsigned EBIAS = 512;

constexpr int L_SLOT = 16384;
constexpr int L_CTL = NR * L_SLOT;
constexpr int L_LANDED = L_CTL;
constexpr int L_CONS = L_CTL + 64;
constexpr int L_MISC = L_CTL + 192;
constexpr int L_BYTES = L_CTL + 256;

struct Params {
  const float* x;
  const float* w;
  float* out;
  char* ws[NR];
  int rank;
  int mode;
};

__device__ __forceinline__ unsigned long long now() { return __builtin_amdgcn_s_memrealtime(); }

__device__ __forceinline__ unsigned pack2(float a, float b) {
  h2 h = {(_Float16)a, (_Float16)b};
  unsigned u = __builtin_bit_cast(unsigned, h);
  return u == 0u ? 0x8000u : u;
}

__device__ __forceinline__ unsigned piece_at(unsigned p) {
  const unsigned s = p >> 6, q = p & 63u;
  const unsigned b = q >> 2, type = (q >> 1) & 1u, half = q & 1u;
  return (type << 8) | (s << 5) | (b * 2u + half);
}

__device__ __forceinline__ void st128(u4 v, __amdgpu_buffer_rsrc_t rs, int voff) {
  __builtin_amdgcn_raw_buffer_store_b128(v, rs, voff, 0, AUX_SYS);
}

__device__ __forceinline__ unsigned lds_ld(const char* smem, int off) {
  return __hip_atomic_load((const unsigned*)(smem + off), __ATOMIC_RELAXED, __HIP_MEMORY_SCOPE_WORKGROUP);
}
__device__ __forceinline__ unsigned long long lds_ld64(const char* smem, int off) {
  return __hip_atomic_load((const unsigned long long*)(smem + off), __ATOMIC_RELAXED, __HIP_MEMORY_SCOPE_WORKGROUP);
}
__device__ __forceinline__ void lds_st(char* smem, int off, unsigned v) {
  __hip_atomic_store((unsigned*)(smem + off), v, __ATOMIC_RELAXED, __HIP_MEMORY_SCOPE_WORKGROUP);
}

#define STAMP(idx) do {} while (0)
#define STAMPV(idx, v) do {} while (0)

__global__ void __launch_bounds__(NWAVE * 64, 1) fused_gemm_ag(Params p) {
  __shared__ __attribute__((aligned(16))) char smem[L_BYTES];
  const int lane = threadIdx.x & 63;
  const int wave = __builtin_amdgcn_readfirstlane(threadIdx.x >> 6);
  const unsigned bid = blockIdx.x;
  const unsigned rank = (unsigned)p.rank & 7u;
  const unsigned xg = bid & 7u, idx = bid >> 3;
  char* const myws = p.ws[0];
  const __amdgpu_buffer_rsrc_t rs_my = __builtin_amdgcn_make_buffer_rsrc((void*)myws, 0, (int)WS_NEED, RSRC3);
  const int ctr_off = (int)(CTR_OFF + (bid * 16u + (unsigned)wave) * 4u);

  if (wave < 2) {
    if (!(p.mode & 1) || bid >= (unsigned)NPUSHWG) return;
  } else {
    if (!(p.mode & 2)) {
      if (wave != 2) return;
      if (lane < 64) lds_st(smem, L_CTL + lane * 4, 0u);
      __builtin_amdgcn_s_waitcnt(0xc07f);
      __builtin_amdgcn_s_barrier();
      return;
    }
  }

  if (wave == 0) {
    __builtin_amdgcn_s_setprio(3);
    STAMP(0);
    const unsigned sigma = xg * 2u + (idx >> 3);
    const unsigned myp = piece_at(sigma + (unsigned)NSTREAM * (unsigned)(lane & (PPS - 1)));
    const __amdgpu_buffer_rsrc_t rs_x = __builtin_amdgcn_make_buffer_rsrc((void*)p.x, 0, MM * KS * 4, RSRC3);
    const __amdgpu_buffer_rsrc_t rs_w = __builtin_amdgcn_make_buffer_rsrc((void*)p.w, 0, KS * NN * 4, RSRC3);
    const __amdgpu_buffer_rsrc_t rs_d = __builtin_amdgcn_make_buffer_rsrc((void*)p.ws[idx & 7u], 0, (int)WS_NEED, RSRC3);
    const int q = lane >> 4, c = lane & 15;
    const int vx = c * (KS * 4) + q * 16;
    const int vw = q * 4 * (NN * 4) + c * 16;
    f4 v[4][2];
    auto issue_loads = [&](unsigned pc) {
      const unsigned type = pc >> 8, s = (pc >> 5) & 7u, g = pc & 31u;
      if (type == 0) {
        const int sb = (int)(g * 64u * (KS * 4) + s * 128u);
#pragma unroll
        for (int a = 0; a < 4; ++a)
#pragma unroll
          for (int b = 0; b < 2; ++b)
            v[a][b] = __builtin_bit_cast(f4, __builtin_amdgcn_raw_buffer_load_b128(rs_x, vx + b * 64, sb + a * 16 * (KS * 4), 0));
      } else {
        const int sb = (int)(s * 32u * (NN * 4) + g * 256u);
#pragma unroll
        for (int a = 0; a < 4; ++a)
#pragma unroll
          for (int b = 0; b < 2; ++b)
            v[a][b] = __builtin_bit_cast(f4, __builtin_amdgcn_raw_buffer_load_b128(rs_w, vw, sb + (a + b * 16) * (NN * 4), 0));
      }
    };
    unsigned pc = __builtin_amdgcn_readlane(myp, 0);
    issue_loads(pc);
    const unsigned call = __builtin_amdgcn_readfirstlane(__builtin_amdgcn_raw_buffer_load_b32(rs_my, ctr_off, 0, AUX_SYS)) + 1u;
    __builtin_amdgcn_s_barrier();
    if (lane == 0) __builtin_amdgcn_raw_buffer_store_b32(call, rs_my, ctr_off, 0, AUX_SYS);
    const unsigned par = call & 1u;
    const unsigned dbase = DATA_OFF + par * PAR_BYTES + rank * SRC_BYTES;
    const unsigned fbase = FLAG_OFF + par * 32768u + rank * 128u;
    for (int n = 0; n < PPS; ++n) {
      const unsigned type = pc >> 8, s = (pc >> 5) & 7u, g = pc & 31u;
      float m = 0.f;
#pragma unroll
      for (int a = 0; a < 4; ++a)
#pragma unroll
        for (int b = 0; b < 2; ++b) {
          m = fmaxf(m, fmaxf(__builtin_fabsf(v[a][b][0]), __builtin_fabsf(v[a][b][1])));
          m = fmaxf(m, fmaxf(__builtin_fabsf(v[a][b][2]), __builtin_fabsf(v[a][b][3])));
        }
      int e = 0;
      const bool big = __builtin_amdgcn_ballot_w64(m >= 32768.f) != 0ull;
      const bool small = __builtin_amdgcn_ballot_w64(m >= 0.00390625f) == 0ull && __builtin_amdgcn_ballot_w64(m > 0.f) != 0ull;
      if (big || small) {
        float mf = 0.f;
#pragma unroll
        for (int a = 0; a < 4; ++a)
#pragma unroll
          for (int b = 0; b < 2; ++b)
#pragma unroll
            for (int t = 0; t < 4; ++t) {
              const float av = __builtin_fabsf(v[a][b][t]);
              mf = fmaxf(mf, av < __builtin_inff() ? av : 0.f);
            }
#pragma unroll
        for (int o = 32; o >= 1; o >>= 1) mf = fmaxf(mf, __shfl_xor(mf, o, 64));
        int ee = 0;
        if (mf > 0.f) { (void)__builtin_frexpf(mf, &ee); ee -= 10; }
        e = __builtin_amdgcn_readfirstlane(ee);
#pragma unroll
        for (int a = 0; a < 4; ++a)
#pragma unroll
          for (int b = 0; b < 2; ++b)
#pragma unroll
            for (int t = 0; t < 4; ++t) v[a][b][t] = __builtin_ldexpf(v[a][b][t], -e);
      }
      u4 o[4];
      if (type == 0) {
#pragma unroll
        for (int a = 0; a < 4; ++a)
          o[a] = u4{pack2(v[a][0][0], v[a][0][1]), pack2(v[a][0][2], v[a][0][3]),
                    pack2(v[a][1][0], v[a][1][1]), pack2(v[a][1][2], v[a][1][3])};
      } else {
#pragma unroll
        for (int i = 0; i < 4; ++i)
          o[i] = u4{pack2(v[0][0][i], v[1][0][i]), pack2(v[2][0][i], v[3][0][i]),
                    pack2(v[0][1][i], v[1][1][i]), pack2(v[2][1][i], v[3][1][i])};
      }
      if (n + 1 < PPS) {
        pc = __builtin_amdgcn_readlane(myp, n + 1);
        issue_loads(pc);
      }
      if (n == PPS - 2) {
        const unsigned long long t0 = now();
        while (lds_ld(smem, L_MISC + 4) == 0u) {
          if (now() - t0 > DEADLINE) break;
          __builtin_amdgcn_s_sleep(2);
        }
      }
      const int dof = (int)(dbase + type * TYPE_BYTES + (s * 32u + g) * 4096u);
#pragma unroll
      for (int i = 0; i < 4; ++i)
        st128(o[i], rs_d, dof + lane * 16 + i * 1024);
      __builtin_amdgcn_raw_buffer_store_b32(((call & CALL_MASK) << 10) | (unsigned)(e + (int)EBIAS), rs_d, lane == 0 ? 0 : 0x40000000,
                                            (int)(fbase + type * 16384u + (g >> 1) * 1024u + s * 8u + (g & 1u) * 4u), AUX_SYS);
      if (n == 0) STAMP(1);
    }
    return;
  }

  if (wave == 1) {
    const unsigned dest = (rank + (idx & 7u)) & 7u;
    const unsigned sigma = xg * 2u + (idx >> 3);
    const unsigned call = __builtin_amdgcn_readfirstlane(__builtin_amdgcn_raw_buffer_load_b32(rs_my, ctr_off, 0, AUX_SYS)) + 1u;
    __builtin_amdgcn_s_barrier();
    if (lane == 0) __builtin_amdgcn_raw_buffer_store_b32(call, rs_my, ctr_off, 0, AUX_SYS);
    const unsigned zbase = DATA_OFF + ((call & 1u) ^ 1u) * PAR_BYTES + dest * SRC_BYTES + sigma * (SRC_BYTES / NSTREAM);
    const u4 z = {0u, 0u, 0u, 0u};
    __builtin_amdgcn_s_sleep(64);
    for (int t = 0; t < 2; ++t)
      for (int u = 0; u < (int)(SRC_BYTES / NSTREAM / 4096); ++u) {
#pragma unroll
        for (int i = 0; i < 4; ++i)
          st128(z, rs_my, (int)(zbase + t * TYPE_BYTES + u * 4096) + lane * 16 + i * 1024);
        __builtin_amdgcn_s_sleep(16);
      }
    asm volatile("s_waitcnt vmcnt(0)" ::: "memory");
    lds_st(smem, L_MISC + 4, 1u);
    STAMP(3);
    return;
  }

  const unsigned ti = (idx >> 3) * 4u + (xg >> 1);
  const unsigned tj = (idx & 7u) * 2u + (xg & 1u);

  if (wave >= 6) {
    const unsigned k = (unsigned)(wave - 6);
    const unsigned src = (rank + k) & 7u;
    const unsigned call = __builtin_amdgcn_readfirstlane(__builtin_amdgcn_raw_buffer_load_b32(rs_my, ctr_off, 0, AUX_SYS)) + 1u;
    __builtin_amdgcn_s_barrier();
    if (lane == 0) __builtin_amdgcn_raw_buffer_store_b32(call, rs_my, ctr_off, 0, AUX_SYS);
    const unsigned par = call & 1u;
    const unsigned tag = call & CALL_MASK;
    const unsigned ltype = (lane >> 1) & 1u;
    const int fo = (int)(FLAG_OFF + par * 32768u + src * 128u + ltype * 16384u + (ltype ? tj : ti) * 1024u + (lane & 1u) * 4u);
    const unsigned dx = DATA_OFF + par * PAR_BYTES + src * SRC_BYTES + ti * 8192u;
    const unsigned dw = DATA_OFF + par * PAR_BYTES + TYPE_BYTES + src * SRC_BYTES + tj * 8192u;
    char* const slot = smem + k * L_SLOT + lane * 16;
    const unsigned long long t0 = now();
    unsigned retries = 0;
    bool dead = false;
    for (unsigned s = 0; s < 8 && !dead; ++s) {
      unsigned got = 0;
      unsigned spins = 0;
      while (got != 3u) {
        asm volatile("" ::: "memory");
        const unsigned f = __builtin_amdgcn_raw_buffer_load_b32(rs_my, fo, (int)(s * 8u), AUX_SYS);
        const unsigned rdy = (unsigned)__builtin_amdgcn_ballot_w64((f >> 10) == tag) & 15u;
        const bool dox = !(got & 1u) && (rdy & 3u) == 3u;
        const bool dow = !(got & 2u) && (rdy & 12u) == 12u;
        if (!(dox || dow)) {
          if ((++spins & 63u) == 0u) {
            const unsigned tm = __builtin_amdgcn_readfirstlane(__builtin_amdgcn_raw_buffer_load_b32(rs_my, (int)TMO_OFF, 0, AUX_SYS));
            if (tm == call || now() - t0 > DEADLINE) { dead = true; break; }
          }
          __builtin_amdgcn_s_sleep(2);
          continue;
        }
        u4 a[8], b[8];
        if (dox) {
#pragma unroll
          for (int i = 0; i < 8; ++i)
            a[i] = __builtin_amdgcn_raw_buffer_load_b128(rs_my, lane * 16 + (i & 3) * 1024, (int)(dx + s * 131072u + (i >> 2) * 4096u), AUX_SYS);
        }
        if (dow) {
#pragma unroll
          for (int i = 0; i < 8; ++i)
            b[i] = __builtin_amdgcn_raw_buffer_load_b128(rs_my, lane * 16 + (i & 3) * 1024, (int)(dw + s * 131072u + (i >> 2) * 4096u), AUX_SYS);
        }
        if (s > 0) {
          unsigned sp2 = 0;
          for (;;) {
            asm volatile("" ::: "memory");
            const unsigned cv = lds_ld(smem, L_CONS + (int)k * 16 + (lane & 3) * 4);
            if (__builtin_amdgcn_ballot_w64(cv < s) == 0ull) break;
            if ((++sp2 & 255u) == 0u && (lds_ld(smem, L_MISC) != 0u || now() - t0 > DEADLINE)) { dead = true; break; }
            __builtin_amdgcn_s_sleep(1);
          }
          if (dead) break;
        }
        if (dox) {
          unsigned mn = 0xFFFFFFFFu;
#pragma unroll
          for (int i = 0; i < 8; ++i) mn = min(min(mn, min(a[i][0], a[i][1])), min(a[i][2], a[i][3]));
          if (__builtin_amdgcn_ballot_w64(mn == 0u) == 0ull) {
#pragma unroll
            for (int i = 0; i < 8; ++i) *(u4*)(slot + i * 1024) = a[i];
            const unsigned e0 = __builtin_amdgcn_readlane(f, 0) & 1023u, e1 = __builtin_amdgcn_readlane(f, 1) & 1023u;
            asm volatile("" ::: "memory");
            lds_st(smem, L_LANDED + (int)k * 8, (s + 1u) | (e0 << 8) | (e1 << 18));
            got |= 1u;
          } else ++retries;
        }
        if (dow) {
          unsigned mn = 0xFFFFFFFFu;
#pragma unroll
          for (int i = 0; i < 8; ++i) mn = min(min(mn, min(b[i][0], b[i][1])), min(b[i][2], b[i][3]));
          if (__builtin_amdgcn_ballot_w64(mn == 0u) == 0ull) {
#pragma unroll
            for (int i = 0; i < 8; ++i) *(u4*)(slot + 8192 + i * 1024) = b[i];
            const unsigned e0 = __builtin_amdgcn_readlane(f, 2) & 1023u, e1 = __builtin_amdgcn_readlane(f, 3) & 1023u;
            asm volatile("" ::: "memory");
            lds_st(smem, L_LANDED + (int)k * 8 + 4, (s + 1u) | (e0 << 8) | (e1 << 18));
            got |= 2u;
          } else ++retries;
        }
        if ((retries & 1023u) == 1023u && now() - t0 > DEADLINE) { dead = true; break; }
      }
    }
    if (dead) {
      if (lane == 0) __builtin_amdgcn_raw_buffer_store_b32(call, rs_my, (int)TMO_OFF, 0, AUX_SYS);
      lds_st(smem, L_MISC, 1u);
    }
    STAMP(8 + k);
    STAMPV(16 + k, retries);
    return;
  }

  const int wk = wave - 2;
  const int wr = wk >> 1, wc = wk & 1;
  if (wk == 0) {
    lds_st(smem, L_CTL + lane * 4, 0u);
    __builtin_amdgcn_s_waitcnt(0xc07f);
  }
  __builtin_amdgcn_s_barrier();
  if (wk == 0) STAMP(4);

  f4 acc[4][4];
#pragma unroll
  for (int a = 0; a < 4; ++a)
#pragma unroll
    for (int b = 0; b < 4; ++b) acc[a][b] = f4{0.f, 0.f, 0.f, 0.f};

  unsigned done = 0;
  bool dead = false;
  const char* const abase = smem + wr * 4096 + lane * 16;
  const char* const bbase = smem + 8192 + wc * 4096 + lane * 16;
  const unsigned long long t0 = now();
  for (int step = 0; step < 64; ++step) {
    unsigned rdy, lx, lw;
    unsigned spins = 0;
    for (;;) {
      asm volatile("" ::: "memory");
      const unsigned long long l = lds_ld64(smem, L_LANDED + (lane & 7) * 8);
      lx = (unsigned)l; lw = (unsigned)(l >> 32);
      rdy = (unsigned)__builtin_amdgcn_ballot_w64((lx & 255u) > done && (lw & 255u) > done) & 255u;
      if (rdy) break;
      if ((++spins & 63u) == 0u && (lds_ld(smem, L_MISC) != 0u || now() - t0 > 2 * DEADLINE)) { dead = true; break; }
      __builtin_amdgcn_s_sleep(1);
    }
    if (dead) break;
    const int k = __builtin_ctz(rdy);
    asm volatile("" ::: "memory");
    h8 af[4], bf[4];
#pragma unroll
    for (int i = 0; i < 4; ++i) bf[i] = *(const h8*)(bbase + k * L_SLOT + i * 1024);
#pragma unroll
    for (int i = 0; i < 4; ++i) af[i] = *(const h8*)(abase + k * L_SLOT + i * 1024);
    asm volatile("" ::: "memory");
    if ((lane & 7) == k) done += 1u;
    if (lane == k) lds_st(smem, L_CONS + k * 16 + wk * 4, done);
    const unsigned sx = __builtin_amdgcn_readlane(lx, k), sw = __builtin_amdgcn_readlane(lw, k);
    const int e = (int)((sx >> (8 + 10 * wr)) & 1023u) + (int)((sw >> (8 + 10 * wc)) & 1023u) - 2 * (int)EBIAS;
    if (e != 0) {
#pragma unroll
      for (int a = 0; a < 4; ++a)
#pragma unroll
        for (int b = 0; b < 4; ++b) {
          const f4 t = __builtin_amdgcn_mfma_f32_16x16x32_f16(af[a], bf[b], f4{0.f, 0.f, 0.f, 0.f}, 0, 0, 0);
#pragma unroll
          for (int r = 0; r < 4; ++r) acc[a][b][r] += __builtin_ldexpf(t[r], e);
          __builtin_amdgcn_sched_barrier(0);
        }
#pragma unroll
      for (int i = 0; i < 4; ++i) { af[i] = h8{}; bf[i] = h8{}; }
    }
#pragma unroll
    for (int a = 0; a < 4; ++a)
#pragma unroll
      for (int b = 0; b < 4; ++b)
        acc[a][b] = __builtin_amdgcn_mfma_f32_16x16x32_f16(af[a], bf[b], acc[a][b], 0, 0, 0);
  }
  if (wk == 0) STAMP(5);

  const __amdgpu_buffer_rsrc_t rs_o = __builtin_amdgcn_make_buffer_rsrc((void*)p.out, 0, MM * NN * 4, RSRC3);
  const float K1 = -2.0f * 0.7978845608028654f * 1.4426950408889634f;
  const float K2 = K1 * 0.044715f;
  const int vo = ((lane >> 4) * 4) * (NN * 4) + (lane & 15) * 16;
  const int so = (int)((ti * 128u + (unsigned)wr * 64u) * (NN * 4) + (tj * 128u + (unsigned)wc * 64u) * 4u);
#pragma unroll
  for (int a = 0; a < 4; ++a)
#pragma unroll
    for (int r = 0; r < 4; ++r) {
      f4 y;
#pragma unroll
      for (int b = 0; b < 4; ++b) {
        const float vv = acc[a][b][r];
        const float ex = __builtin_amdgcn_exp2f(vv * __builtin_fmaf(vv * vv, K2, K1));
        y[b] = vv * __builtin_amdgcn_rcpf(1.0f + ex);
      }
      if (dead) y = f4{__builtin_nanf(""), __builtin_nanf(""), __builtin_nanf(""), __builtin_nanf("")};
      st128(__builtin_bit_cast(u4, y), rs_o, vo + so + (a * 16 + r) * (NN * 4));
    }
}

}

extern "C" void kernel_launch(int rank, int nranks, ncclComm_t comm,
                              hipStream_t stream,
                              void* const* d_in, const int* in_sizes, int n_in,
                              void* d_out, int out_size,
                              const int* in_dtypes, int out_dtype,
                              void* const* d_out_all,
                              void* const* d_ws_all, size_t ws_bytes) {
  (void)comm; (void)in_dtypes; (void)out_dtype; (void)d_out_all;
  if (nranks != NR || (unsigned)rank >= (unsigned)NR || n_in < 2 || ws_bytes < (size_t)WS_NEED ||
      in_sizes[0] != MM * KS || in_sizes[1] != KS * NN || out_size != MM * NN) {
    fprintf(stderr, "kernel_launch: unexpected problem shape\n");
    return;
  }
  static int ncu = 0;
  if (ncu == 0) {
    int dev = 0, n = 0;
    HIP_CHECK(hipGetDevice(&dev));
    HIP_CHECK(hipDeviceGetAttribute(&n, hipDeviceAttributeMultiprocessorCount, dev));
    ncu = n > 0 ? n : 1;
  }
  Params p;
  p.x = (const float*)d_in[0];
  p.w = (const float*)d_in[1];
  p.out = (float*)d_out;
  for (int r = 0; r < NR; ++r) p.ws[r] = (char*)d_ws_all[(rank + r) % NR];
  p.rank = rank;
  if (ncu >= NWG) {
    p.mode = 3;
    hipLaunchKernelGGL(fused_gemm_ag, dim3(NWG), dim3(NWAVE * 64), 0, stream, p);
  } else {
    p.mode = 1;
    hipLaunchKernelGGL(fused_gemm_ag, dim3(NWG), dim3(NWAVE * 64), 0, stream, p);
    p.mode = 2;
    hipLaunchKernelGGL(fused_gemm_ag, dim3(NWG), dim3(NWAVE * 64), 0, stream, p);
  }
  HIP_CHECK(hipGetLastError());
}
```

```cpp
#include <hip/hip_runtime.h>
#include <rccl/rccl.h>
#include <cstdio>
#include <cstdint>

#define HIP_CHECK(expr) do { hipError_t _e = (expr); if (_e != hipSuccess) { \
    fprintf(stderr, "HIP error %s at %s:%d\n", hipGetErrorString(_e), __FILE__, __LINE__); \
    return; } } while (0)

typedef _Float16 h8 __attribute__((ext_vector_type(8)));
typedef _Float16 h2 __attribute__((ext_vector_type(2)));
typedef float f4 __attribute__((ext_vector_type(4)));
typedef float f2 __attribute__((ext_vector_type(2)));
typedef unsigned u4 __attribute__((ext_vector_type(4)));
typedef unsigned u2 __attribute__((ext_vector_type(2)));

#define SLIDE 4
#define CONV_SHIFT 4
#define CONV_SLEEP 4
#define FETCH_DELAY 40
#define FETCH_AHEAD 2
#define AUX_POLL 17
#define POLL_SLEEP 2

namespace {

constexpr int NR = 8;
constexpr int MM = 2048, NN = 2048, KS = 256;
constexpr int NWG = 256;
constexpr int NPW = 16;
constexpr int NGW = NWG - NPW;
constexpr int NWAVE = 12;
constexpr int NCONV = 5;
constexpr int NPIECE = 512;
constexpr int PPW = NPIECE / NPW;

constexpr unsigned CTR_OFF  = 0;
constexpr unsigned TMO_OFF  = 32768;
constexpr unsigned TKT_OFF  = 36992;
constexpr unsigned ZCNT_OFF = 36864;
constexpr unsigned FLAG_OFF = 65536;
constexpr unsigned DBG_OFF  = 262144;
constexpr unsigned DATA_OFF = 1u << 20;
constexpr unsigned PAR_BYTES = 16u << 20;
constexpr unsigned TYPE_BYTES = 8u << 20;
constexpr unsigned SRC_BYTES = 1u << 20;
constexpr unsigned WS_NEED = DATA_OFF + 2 * PAR_BYTES;

constexpr int AUX_SYS = 17;
constexpr unsigned RSRC3 = 0x00020000u;
constexpr unsigned long long DEADLINE = 100000000ull;
constexpr unsigned CALL_MASK = 0x3FFFFFu;
constexpr unsigned EBIAS = 512;
constexpr int OOR = 0x40000000;

constexpr int L_SLOT = 17408;
constexpr int L_CTL = NR * L_SLOT;
constexpr int L_LANDED = L_CTL;
constexpr int L_CONS = L_CTL + 128;
constexpr int L_MISC = L_CTL + 256;
constexpr int L_BYTES = L_CTL + 512;
constexpr int RING = 16;
constexpr int L_RDY = RING * 4096;
constexpr int L_PROG = L_CTL;
constexpr unsigned R_END = 0xFFFFu, R_SKIP = 0xFFFEu;

struct Params {
  const float* x;
  const float* w;
  float* out;
  char* ws[NR];
  int rank;
  int mode;
};

__device__ __forceinline__ unsigned long long now() { return __builtin_amdgcn_s_memrealtime(); }

__device__ __forceinline__ unsigned pack2(float a, float b) {
  h2 h = {(_Float16)a, (_Float16)b};
  return __builtin_bit_cast(unsigned, h);
}
__device__ __forceinline__ float max3abs(float a, float b, float m) {
  float r;
  asm("v_max3_f32 %0, |%1|, |%2|, %3" : "=v"(r) : "v"(a), "v"(b), "v"(m));
  return r;
}

__device__ __forceinline__ unsigned piece_at(unsigned p) {
  const unsigned s = p >> 6, q = p & 63u;
  const unsigned type = (q >> 1) & 1u, half = q & 1u;
  const unsigned b = ((q >> 2) + 15u + type) & 15u;
  return (type << 8) | (s << 5) | (b * 2u + half);
}

__device__ __forceinline__ void st128(u4 v, __amdgpu_buffer_rsrc_t rs, int voff) {
  __builtin_amdgcn_raw_buffer_store_b128(v, rs, voff, 0, AUX_SYS);
}

__device__ __forceinline__ unsigned lds_ld(const char* smem, int off) {
  return __hip_atomic_load((const unsigned*)(smem + off), __ATOMIC_RELAXED, __HIP_MEMORY_SCOPE_WORKGROUP);
}
__device__ __forceinline__ unsigned long long lds_ld64(const char* smem, int off) {
  return __hip_atomic_load((const unsigned long long*)(smem + off), __ATOMIC_RELAXED, __HIP_MEMORY_SCOPE_WORKGROUP);
}
__device__ __forceinline__ void lds_st(char* smem, int off, unsigned v) {
  __hip_atomic_store((unsigned*)(smem + off), v, __ATOMIC_RELAXED, __HIP_MEMORY_SCOPE_WORKGROUP);
}
__device__ __forceinline__ void lds_st64(char* smem, int off, unsigned long long v) {
  __hip_atomic_store((unsigned long long*)(smem + off), v, __ATOMIC_RELAXED, __HIP_MEMORY_SCOPE_WORKGROUP);
}
__device__ __forceinline__ unsigned min4(u4 v, unsigned m) { return min(min(m, min(v[0], v[1])), min(v[2], v[3])); }

#define STAMP(idx) do {} while (0)
#define STAMPV(idx, v) do {} while (0)

__global__ void __launch_bounds__(NWAVE * 64, 1) fused_gemm_ag(Params p) {
  __shared__ __attribute__((aligned(16))) char smem[L_BYTES];
  const int lane = threadIdx.x & 63;
  const int wave = __builtin_amdgcn_readfirstlane(threadIdx.x >> 6);
  const unsigned bid = blockIdx.x;
  const unsigned rank = (unsigned)p.rank & 7u;
  char* const myws = p.ws[0];
  const __amdgpu_buffer_rsrc_t rs_my = __builtin_amdgcn_make_buffer_rsrc((void*)myws, 0, (int)WS_NEED, RSRC3);
  const int ctr_off = (int)(CTR_OFF + (bid * 16u + (unsigned)wave) * 4u);
  const int lane0 = lane == 0 ? 0 : OOR;

  const bool sendwg = bid < (unsigned)NPW;
  if (sendwg ? !(p.mode & 1) : (wave >= 4 && !(p.mode & 2))) return;

  const bool conv = (p.mode & 1) && wave < 2;
  const unsigned cpos = sendwg ? (unsigned)(wave & 1) * NPW + bid : 2u * NPW + (unsigned)(wave & 1) * NGW + (bid - NPW);
  const unsigned cpc = piece_at(cpos & (NPIECE - 1));
  f4 R[4][2];
  auto conv_issue = [&]() {
    const int q = lane >> 4, c = lane & 15;
    const bool isw = (cpc >> 8) != 0u;
    const unsigned s = (cpc >> 5) & 7u, g = cpc & 31u;
    const __amdgpu_buffer_rsrc_t rs = __builtin_amdgcn_make_buffer_rsrc((void*)(isw ? p.w : p.x), 0, MM * KS * 4, RSRC3);
    const int sb = isw ? (int)(s * 32u * (NN * 4) + g * 256u) : (int)(g * 64u * (KS * 4) + s * 128u);
    const int sa = isw ? NN * 4 : 16 * KS * 4;
    const int sbb = isw ? 16 * NN * 4 : 64;
    const int vo = isw ? q * 4 * (NN * 4) + c * 16 : c * (KS * 4) + q * 16;
#pragma unroll
    for (int a = 0; a < 4; ++a)
#pragma unroll
      for (int b = 0; b < 2; ++b)
        R[a][b] = __builtin_bit_cast(f4, __builtin_amdgcn_raw_buffer_load_b128(rs, vo, sb + a * sa + b * sbb, 0));
  };
  if (conv && sendwg) conv_issue();
  const bool sender = sendwg && wave >= NCONV;
  const unsigned cv = __builtin_amdgcn_raw_buffer_load_b32(rs_my, sender ? OOR : ctr_off, 0, AUX_SYS);
  if (wave == 2) {
    lds_st(smem, L_CTL + lane * 4, 0u);
    lds_st(smem, L_CTL + 256 + lane * 4, 0u);
  }
  if (sendwg && wave >= 2 && wave < 7) *(u4*)(smem + L_RDY + (wave - 2) * 1024 + lane * 16) = u4{0u, 0u, 0u, 0u};
  __builtin_amdgcn_s_waitcnt(0xc07f);
  __builtin_amdgcn_s_barrier();
  const unsigned call = __builtin_amdgcn_readfirstlane(cv) + 1u;
  const unsigned par = call & 1u;
  const unsigned tag = call & CALL_MASK;
  if (!sender && (wave >= 4 || (p.mode & 1))) __builtin_amdgcn_raw_buffer_store_b32(call, rs_my, lane0, ctr_off, AUX_SYS);
  auto data_off = [&](unsigned pc, unsigned pr) -> unsigned {
    return DATA_OFF + pr * PAR_BYTES + rank * SRC_BYTES + (pc >> 8) * TYPE_BYTES + (pc & 255u) * 4096u;
  };
  auto flag_off = [&](unsigned pc, unsigned pr) -> unsigned {
    return FLAG_OFF + pr * 32768u + rank * 128u + (pc >> 8) * 16384u + ((pc >> 1) & 15u) * 1024u + ((pc >> 5) & 7u) * 8u + (pc & 1u) * 4u;
  };
  if (conv) {
    if (!sendwg) {
      for (unsigned i = 0; i < (cpos >> CONV_SHIFT); ++i) __builtin_amdgcn_s_sleep(CONV_SLEEP);
      conv_issue();
    }
    float m = 0.f;
#pragma unroll
    for (int a = 0; a < 4; ++a)
#pragma unroll
      for (int b = 0; b < 2; ++b) {
        m = max3abs(R[a][b][0], R[a][b][1], m);
        m = max3abs(R[a][b][2], R[a][b][3], m);
      }
    int e = 0;
    const bool big = __builtin_amdgcn_ballot_w64(m >= 32768.f) != 0ull;
    const bool small = __builtin_amdgcn_ballot_w64(m >= 0.00390625f) == 0ull && __builtin_amdgcn_ballot_w64(m > 0.f) != 0ull;
    if (big || small) {
      float mf = 0.f;
#pragma unroll
      for (int a = 0; a < 4; ++a)
#pragma unroll
        for (int b = 0; b < 2; ++b)
#pragma unroll
          for (int t = 0; t < 4; ++t) {
            const float av = __builtin_fabsf(R[a][b][t]);
            mf = fmaxf(mf, av < __builtin_inff() ? av : 0.f);
          }
#pragma unroll
      for (int o = 32; o >= 1; o >>= 1) mf = fmaxf(mf, __shfl_xor(mf, o, 64));
      int ee = 0;
      if (mf > 0.f) { (void)__builtin_frexpf(mf, &ee); ee -= 10; }
      e = __builtin_amdgcn_readfirstlane(ee);
#pragma unroll
      for (int a = 0; a < 4; ++a)
#pragma unroll
        for (int b = 0; b < 2; ++b)
#pragma unroll
          for (int t = 0; t < 4; ++t) R[a][b][t] = __builtin_ldexpf(R[a][b][t], -e);
    }
    u4 o[4];
    if ((cpc >> 8) == 0u) {
#pragma unroll
      for (int a = 0; a < 4; ++a)
        o[a] = u4{pack2(R[a][0][0], R[a][0][1]), pack2(R[a][0][2], R[a][0][3]),
                  pack2(R[a][1][0], R[a][1][1]), pack2(R[a][1][2], R[a][1][3])};
    } else {
#pragma unroll
      for (int i = 0; i < 4; ++i)
        o[i] = u4{pack2(R[0][0][i], R[1][0][i]), pack2(R[2][0][i], R[3][0][i]),
                  pack2(R[0][1][i], R[1][1][i]), pack2(R[2][1][i], R[3][1][i])};
    }
    unsigned mn = 0xFFFFFFFFu;
#pragma unroll
    for (int i = 0; i < 4; ++i) mn = min4(o[i], mn);
    if (__builtin_amdgcn_ballot_w64(mn == 0u) != 0ull) {
#pragma unroll
      for (int i = 0; i < 4; ++i)
#pragma unroll
        for (int t = 0; t < 4; ++t) o[i][t] = o[i][t] == 0u ? 0x8000u : o[i][t];
    }
    const unsigned h = (tag << 10) | (unsigned)(e + (int)EBIAS);
    if (sendwg) {
#pragma unroll
      for (int i = 0; i < 4; ++i) *(u4*)(smem + wave * 4096 + i * 1024 + lane * 16) = o[i];
      asm volatile("" ::: "memory");
      lds_st64(smem, L_RDY + wave * 8, (unsigned long long)h | ((unsigned long long)cpos << 32));
      if (wave == 0) STAMP(5);
      asm volatile("" ::: "memory");
    }
    const int dof = (int)data_off(cpc, par) + lane * 16;
#pragma unroll
    for (int i = 0; i < 4; ++i) st128(o[i], rs_my, dof + i * 1024);
    __builtin_amdgcn_raw_buffer_store_b32(h, rs_my, lane0, (int)flag_off(cpc, par), AUX_SYS);
  }

  if (sendwg) {
    const unsigned long long t0 = now();
    if (sender) {
      const int dk = wave - (NCONV - 1);
      __builtin_amdgcn_s_setprio(3);
      const __amdgpu_buffer_rsrc_t rs_d = __builtin_amdgcn_make_buffer_rsrc((void*)p.ws[dk], 0, (int)WS_NEED, RSRC3);
      unsigned ends = 0;
      for (unsigned n = 0; ends < (unsigned)NCONV; ++n) {
        unsigned h, pos, spins = 0;
        for (;;) {
          asm volatile("" ::: "memory");
          const unsigned long long r = lds_ld64(smem, L_RDY + (int)n * 8);
          h = __builtin_amdgcn_readfirstlane((unsigned)r);
          pos = __builtin_amdgcn_readfirstlane((unsigned)(r >> 32));
          if (h != 0u) break;
          if ((++spins & 255u) == 0u && now() - t0 > 2 * DEADLINE) return;
          __builtin_amdgcn_s_sleep(1);
        }
        asm volatile("" ::: "memory");
        if (pos >= (unsigned)NPIECE) {
          ends += pos == R_END ? 1u : 0u;
          lds_st(smem, L_PROG + (dk - 1) * 4, n + 1u);
          continue;
        }
        u4 o[4];
#pragma unroll
        for (int i = 0; i < 4; ++i) o[i] = *(const u4*)(smem + (n & (RING - 1)) * 4096u + i * 1024 + lane * 16);
        asm volatile("" ::: "memory");
        lds_st(smem, L_PROG + (dk - 1) * 4, n + 1u);
        const unsigned pc = piece_at(pos);
        const int dof = (int)data_off(pc, (h >> 10) & 1u) + lane * 16;
#pragma unroll
        for (int i = 0; i < 4; ++i) {
          asm volatile("s_waitcnt vmcnt(%0)" :: "n"(SLIDE) : "memory");
          st128(o[i], rs_d, dof + i * 1024);
        }
        __builtin_amdgcn_raw_buffer_store_b32(h, rs_d, lane0, (int)flag_off(pc, (h >> 10) & 1u), AUX_SYS);
      }
      return;
    }
    if (wave >= 2) __builtin_amdgcn_s_sleep(FETCH_DELAY);
    for (unsigned n = (unsigned)wave + (wave < 2 ? NCONV : 0); ; n += NCONV) {
      unsigned pos = n * NPW + bid;
      unsigned spins = 0;
      if (n >= (unsigned)NCONV) {
        for (;;) {
          asm volatile("" ::: "memory");
          const unsigned pr = lds_ld(smem, L_PROG + (lane < NR - 1 ? lane : 0) * 4);
          if (__builtin_amdgcn_ballot_w64(pr + FETCH_AHEAD < n) == 0ull) break;
          if ((++spins & 255u) == 0u && now() - t0 > 2 * DEADLINE) return;
          __builtin_amdgcn_s_sleep(2);
        }
        const int raw = __builtin_amdgcn_raw_ptr_buffer_atomic_add_i32(1, rs_my, lane0, (int)TKT_OFF, 0);
        pos = ((unsigned)__builtin_amdgcn_readfirstlane(raw) & (NPIECE - 1)) + NCONV * NPW;
        if (pos >= (unsigned)NPIECE) {
          lds_st64(smem, L_RDY + (int)(n + NCONV) * 8, 1ull | ((unsigned long long)R_SKIP << 32));
          lds_st64(smem, L_RDY + (int)n * 8, 1ull | ((unsigned long long)R_END << 32));
          break;
        }
      }
      const unsigned pc = piece_at(pos);
      for (;;) {
        u4 o[4];
#pragma unroll
        for (int i = 0; i < 4; ++i)
          o[i] = __builtin_amdgcn_raw_buffer_load_b128(rs_my, lane * 16 + i * 1024, (int)data_off(pc, par), AUX_SYS);
        const unsigned h = __builtin_amdgcn_raw_buffer_load_b32(rs_my, 0, (int)flag_off(pc, par), AUX_SYS);
        unsigned mn = (h >> 10) == tag ? 0xFFFFFFFFu : 0u;
        if (pos == (unsigned)NPIECE - 1u) {
          const unsigned z = __builtin_amdgcn_raw_buffer_load_b32(rs_my, 0, (int)ZCNT_OFF, AUX_SYS);
          if ((int)(z - call * (unsigned)NGW) < 0) mn = 0u;
        }
#pragma unroll
        for (int i = 0; i < 4; ++i) mn = min4(o[i], mn);
        if (__builtin_amdgcn_ballot_w64(mn == 0u) == 0ull) {
#pragma unroll
          for (int i = 0; i < 4; ++i) *(u4*)(smem + (n & (RING - 1)) * 4096u + i * 1024 + lane * 16) = o[i];
          asm volatile("" ::: "memory");
          lds_st64(smem, L_RDY + (int)n * 8, (unsigned long long)h | ((unsigned long long)pos << 32));
          break;
        }
        if ((++spins & 63u) == 0u && now() - t0 > DEADLINE) return;
        __builtin_amdgcn_s_sleep(4);
      }
    }
    if (wave == 0) { STAMP(8); STAMPV(0, ts00); STAMPV(6, ts0); }
    return;
  }

  const unsigned gi = (bid >> 3) - (unsigned)(NPW / 8);
  const unsigned ti = gi >> 1;
  const unsigned tj = (bid & 7u) * 2u + (gi & 1u);
  const bool hasx = ti < 8u;

  if (wave >= 4) {
    const unsigned k = (unsigned)(wave - 4);
    const unsigned src = (rank + k) & 7u;
    const unsigned ltype = (lane >> 1) & 1u;
    const int fo = (int)(FLAG_OFF + par * 32768u + src * 128u +
                         (lane == 4 && hasx ? 15u * 1024u + (ti >> 2) * 4u : ltype * 16384u + (ltype ? tj : ti) * 1024u + (lane & 1u) * 4u));
    const unsigned dx = DATA_OFF + par * PAR_BYTES + src * SRC_BYTES + ti * 8192u;
    const unsigned de = DATA_OFF + par * PAR_BYTES + src * SRC_BYTES + 15u * 8192u + ti * 1024u;
    const unsigned dw = DATA_OFF + par * PAR_BYTES + TYPE_BYTES + src * SRC_BYTES + tj * 8192u;
    char* const slot = smem + k * L_SLOT + lane * 16;
    const unsigned long long t0 = now();
    unsigned retries = 0;
    bool dead = false;
    for (unsigned s = 0; s < 8 && !dead; ++s) {
      unsigned got = 0;
      unsigned spins = 0;
      while (got != 3u) {
        asm volatile("" ::: "memory");
        const unsigned f = __builtin_amdgcn_raw_buffer_load_b32(rs_my, fo, (int)(s * 8u), AUX_POLL);
        const unsigned rdy = (unsigned)__builtin_amdgcn_ballot_w64((f >> 10) == tag);
        const bool dox = !(got & 1u) && (rdy & 19u) == 19u;
        const bool dow = !(got & 2u) && (rdy & 12u) == 12u;
        if (!(dox || dow)) {
          if ((++spins & 63u) == 0u) {
            const unsigned tm = __builtin_amdgcn_readfirstlane(__builtin_amdgcn_raw_buffer_load_b32(rs_my, (int)TMO_OFF, 0, AUX_SYS));
            if (tm == call || now() - t0 > DEADLINE) { dead = true; break; }
          }
          __builtin_amdgcn_s_sleep(POLL_SLEEP);
          continue;
        }
        u4 a[8], b[8], ae = u4{1u, 1u, 1u, 1u};
        if (dox) {
#pragma unroll
          for (int i = 0; i < 8; ++i)
            a[i] = __builtin_amdgcn_raw_buffer_load_b128(rs_my, lane * 16 + (i & 3) * 1024, (int)(dx + s * 131072u + (i >> 2) * 4096u), AUX_SYS);
          if (hasx) ae = __builtin_amdgcn_raw_buffer_load_b128(rs_my, lane * 16, (int)(de + s * 131072u), AUX_SYS);
        }
        if (dow) {
#pragma unroll
          for (int i = 0; i < 8; ++i)
            b[i] = __builtin_amdgcn_raw_buffer_load_b128(rs_my, lane * 16 + (i & 3) * 1024, (int)(dw + s * 131072u + (i >> 2) * 4096u), AUX_SYS);
        }
        if (s > 0) {
          unsigned sp2 = 0;
          for (;;) {
            asm volatile("" ::: "memory");
            const unsigned cv = lds_ld(smem, L_CONS + (int)k * 16 + (lane & 3) * 4);
            if (__builtin_amdgcn_ballot_w64(cv < s) == 0ull) break;
            if ((++sp2 & 255u) == 0u && (lds_ld(smem, L_MISC) != 0u || now() - t0 > DEADLINE)) { dead = true; break; }
            __builtin_amdgcn_s_sleep(1);
          }
          if (dead) break;
        }
        if (dox) {
          unsigned mn = min4(ae, 0xFFFFFFFFu);
#pragma unroll
          for (int i = 0; i < 8; ++i) mn = min4(a[i], mn);
          if (__builtin_amdgcn_ballot_w64(mn == 0u) == 0ull) {
#pragma unroll
            for (int i = 0; i < 8; ++i) *(u4*)(slot + i * 1024) = a[i];
            *(u4*)(slot + 16384) = ae;
            const unsigned e0 = __builtin_amdgcn_readlane(f, 0) & 1023u, e1 = __builtin_amdgcn_readlane(f, 1) & 1023u;
            const unsigned e2 = __builtin_amdgcn_readlane(f, 4) & 1023u;
            asm volatile("" ::: "memory");
            lds_st64(smem, L_LANDED + (int)k * 16, (unsigned long long)((s + 1u) | (e0 << 8) | (e1 << 18)) | ((unsigned long long)e2 << 32));
            got |= 1u;
          } else ++retries;
        }
        if (dow) {
          unsigned mn = 0xFFFFFFFFu;
#pragma unroll
          for (int i = 0; i < 8; ++i) mn = min4(b[i], mn);
          if (__builtin_amdgcn_ballot_w64(mn == 0u) == 0ull) {
#pragma unroll
            for (int i = 0; i < 8; ++i) *(u4*)(slot + 8192 + i * 1024) = b[i];
            const unsigned e0 = __builtin_amdgcn_readlane(f, 2) & 1023u, e1 = __builtin_amdgcn_readlane(f, 3) & 1023u;
            asm volatile("" ::: "memory");
            lds_st(smem, L_LANDED + (int)k * 16 + 8, (s + 1u) | (e0 << 8) | (e1 << 18));
            got |= 2u;
          } else ++retries;
        }
        if ((retries & 1023u) == 1023u && now() - t0 > DEADLINE) { dead = true; break; }
      }
    }
    if (dead) {
      __builtin_amdgcn_raw_buffer_store_b32(call, rs_my, lane0, (int)TMO_OFF, AUX_SYS);
      lds_st(smem, L_MISC, 1u);
    }
    STAMP(8 + k);
    STAMPV(16 + k, retries);
    return;
  }

  const int wk = wave;
  const int wr = wk >> 1, wc = wk & 1;
  if (wk == 0) STAMPV(4, ts00);
  if (wk == 0) STAMP(2);
  const __amdgpu_buffer_rsrc_t rs_z = __builtin_amdgcn_make_buffer_rsrc(
      (void*)(myws + DATA_OFF + (par ^ 1u) * PAR_BYTES), 0, (p.mode & 1) ? (int)PAR_BYTES : 0, RSRC3);
  const int zo = (int)(((bid - NPW) * 4u + (unsigned)wk) * 1024u) + lane * 16;
  constexpr int ZSTEPS = (int)((PAR_BYTES / 1024 + NGW * 4 - 1) / (NGW * 4));
  static_assert(ZSTEPS % 2 == 0 && ZSTEPS <= 32, "clearing schedule");
  auto zero_done = [&]() {
    asm volatile("s_waitcnt vmcnt(0)" ::: "memory");
    unsigned old = 0;
    if (lane == 0) old = __hip_atomic_fetch_add((unsigned*)(smem + L_MISC + 4), 1u, __ATOMIC_RELAXED, __HIP_MEMORY_SCOPE_WORKGROUP);
    if (__builtin_amdgcn_readfirstlane(old) == 3u)
      (void)__builtin_amdgcn_raw_ptr_buffer_atomic_add_i32(1, rs_my, lane0, (int)ZCNT_OFF, 0);
  };
  if (!(p.mode & 2)) {
    for (int i = 0; i < ZSTEPS; ++i) st128(u4{0u, 0u, 0u, 0u}, rs_z, zo + i * (NGW * 4 * 1024));
    zero_done();
    return;
  }

  float* outp = p.out;
  asm volatile("" : "+s"(outp));
  f4 acc[4][4], acx[2];
#pragma unroll
  for (int a = 0; a < 4; ++a)
#pragma unroll
    for (int b = 0; b < 4; ++b) acc[a][b] = f4{0.f, 0.f, 0.f, 0.f};
  acx[0] = acx[1] = f4{0.f, 0.f, 0.f, 0.f};

  unsigned done = 0;
  bool dead = false;
  const char* const abase = smem + wr * 4096 + lane * 16;
  const char* const bbase = smem + 8192 + wc * 4096 + lane * 16;
  const char* const ebase = smem + 16384 + lane * 16;
  const int xsel = hasx ? wr + 1 : 0;
  const unsigned long long t0 = now();
  struct Frag { h8 b[4], a[2], e; };
  h8 af[2];
  Frag f0, f1;
  int e = 0, e2 = 0;
  unsigned lx, le, lw;
  auto peek = [&]() {
    asm volatile("" ::: "memory");
    const unsigned long long l = lds_ld64(smem, L_LANDED + (lane & 7) * 16);
    lw = lds_ld(smem, L_LANDED + (lane & 7) * 16 + 8);
    lx = (unsigned)l; le = (unsigned)(l >> 32);
  };
  auto ready = [&]() -> unsigned {
    return (unsigned)__builtin_amdgcn_ballot_w64((lx & 255u) > done && (lw & 255u) > done) & 255u;
  };
  auto read = [&](Frag& fn, int k) {
    asm volatile("" ::: "memory");
#pragma unroll
    for (int i = 0; i < 4; ++i) fn.b[i] = *(const h8*)(bbase + k * L_SLOT + i * 1024);
#pragma unroll
    for (int i = 0; i < 2; ++i) af[i] = *(const h8*)(abase + k * L_SLOT + i * 1024);
#pragma unroll
    for (int i = 0; i < 2; ++i) fn.a[i] = *(const h8*)(abase + k * L_SLOT + 2048 + i * 1024);
    fn.e = *(const h8*)(ebase + k * L_SLOT);
    asm volatile("" ::: "memory");
  };
  auto take = [&](int k) {
    if ((lane & 7) == k) done += 1u;
    if (lane == k) lds_st(smem, L_CONS + k * 16 + wk * 4, done);
    const unsigned sx = __builtin_amdgcn_readlane(lx, k), se = __builtin_amdgcn_readlane(le, k), sw = __builtin_amdgcn_readlane(lw, k);
    const int ew = (int)((sw >> (8 + 10 * wc)) & 1023u) - 2 * (int)EBIAS;
    e = (int)((sx >> (8 + 10 * wr)) & 1023u) + ew;
    e2 = (int)se + ew;
  };
  auto wait_and_read = [&](Frag& fn) {
    unsigned rdy, spins = 0;
    for (;;) {
      peek();
      rdy = ready();
      if (rdy) break;
      if ((++spins & 63u) == 0u && (lds_ld(smem, L_MISC) != 0u || now() - t0 > 2 * DEADLINE)) { dead = true; rdy = 1u; break; }
      __builtin_amdgcn_s_sleep(1);
    }
    const int k = __builtin_ctz(rdy);
    read(fn, k);
    take(k);
  };
  auto body = [&](Frag& f, Frag& fn, int step) {
    peek();
    if (step < ZSTEPS / 2) {
      st128(u4{0u, 0u, 0u, 0u}, rs_z, zo + step * (NGW * 8 * 1024));
      st128(u4{0u, 0u, 0u, 0u}, rs_z, zo + step * (NGW * 8 * 1024) + NGW * 4 * 1024);
    }
    if (step == 16 && (p.mode & 1)) zero_done();
    if ((e | e2) != 0) {
#pragma unroll
      for (int a = 0; a < 4; ++a)
#pragma unroll
        for (int b = 0; b < 4; ++b) {
          const f4 t = __builtin_amdgcn_mfma_f32_16x16x32_f16(a < 2 ? af[a & 1] : f.a[a & 1], f.b[b], f4{0.f, 0.f, 0.f, 0.f}, 0, 0, 0);
#pragma unroll
          for (int r = 0; r < 4; ++r) acc[a][b][r] += __builtin_ldexpf(t[r], e);
          __builtin_amdgcn_sched_barrier(0);
        }
      const f4 t0 = __builtin_amdgcn_mfma_f32_16x16x32_f16(f.e, wr ? f.b[2] : f.b[0], f4{0.f, 0.f, 0.f, 0.f}, 0, 0, 0);
      const f4 t1 = __builtin_amdgcn_mfma_f32_16x16x32_f16(f.e, wr ? f.b[3] : f.b[1], f4{0.f, 0.f, 0.f, 0.f}, 0, 0, 0);
#pragma unroll
      for (int r = 0; r < 4; ++r) { acx[0][r] += __builtin_ldexpf(t0[r], e2); acx[1][r] += __builtin_ldexpf(t1[r], e2); }
#pragma unroll
      for (int i = 0; i < 4; ++i) f.b[i] = h8{};
      af[0] = af[1] = f.a[0] = f.a[1] = f.e = h8{};
    }
#pragma unroll
    for (int a = 0; a < 2; ++a)
#pragma unroll
      for (int b = 0; b < 4; ++b)
        acc[a][b] = __builtin_amdgcn_mfma_f32_16x16x32_f16(af[a], f.b[b], acc[a][b], 0, 0, 0);
    __builtin_amdgcn_sched_barrier(0);
    const unsigned rdy = ready();
    const int k = __builtin_ctz(rdy | 256u) & 7;
    read(fn, k);
    __builtin_amdgcn_sched_barrier(0);
#pragma unroll
    for (int a = 2; a < 4; ++a)
#pragma unroll
      for (int b = 0; b < 4; ++b)
        acc[a][b] = __builtin_amdgcn_mfma_f32_16x16x32_f16(f.a[a - 2], f.b[b], acc[a][b], 0, 0, 0);
    if (xsel == 1) {
      acx[0] = __builtin_amdgcn_mfma_f32_16x16x32_f16(f.e, f.b[0], acx[0], 0, 0, 0);
      acx[1] = __builtin_amdgcn_mfma_f32_16x16x32_f16(f.e, f.b[1], acx[1], 0, 0, 0);
      asm volatile("" ::: "memory");
    }
    if (xsel == 2) {
      acx[0] = __builtin_amdgcn_mfma_f32_16x16x32_f16(f.e, f.b[2], acx[0], 0, 0, 0);
      acx[1] = __builtin_amdgcn_mfma_f32_16x16x32_f16(f.e, f.b[3], acx[1], 0, 0, 0);
      asm volatile("" ::: "memory");
    }
    __builtin_amdgcn_sched_barrier(0);
    if (rdy) take(k);
    else if (step < 63) wait_and_read(fn);
  };
  wait_and_read(f0);
  for (int step = 0; step < 64 && !dead; step += 2) {
    body(f0, f1, step);
    body(f1, f0, step + 1);
  }
  if (wk == 0) STAMP(5);
  if (dead) {
    const float qn = __builtin_nanf("");
#pragma unroll
    for (int a = 0; a < 4; ++a)
#pragma unroll
      for (int b = 0; b < 4; ++b) acc[a][b] = f4{qn, qn, qn, qn};
    acx[0] = acx[1] = f4{qn, qn, qn, qn};
  }

  const __amdgpu_buffer_rsrc_t rs_o = __builtin_amdgcn_make_buffer_rsrc((void*)outp, 0, MM * NN * 4, RSRC3);
  const float K1 = -2.0f * 0.7978845608028654f * 1.4426950408889634f;
  const float K2 = K1 * 0.044715f;
  auto gelu2 = [&](f2 v) -> f2 {
    const f2 t = v * __builtin_elementwise_fma(v * v, f2{K2, K2}, f2{K1, K1});
    const f2 d = f2{__builtin_amdgcn_exp2f(t[0]), __builtin_amdgcn_exp2f(t[1])} + f2{1.f, 1.f};
    return v * f2{__builtin_amdgcn_rcpf(d[0]), __builtin_amdgcn_rcpf(d[1])};
  };
  const int vo = ((lane >> 4) * 4) * (NN * 4) + (lane & 15) * 16;
  const int so = (int)((ti * 128u + (unsigned)wr * 64u) * (NN * 4) + (tj * 128u + (unsigned)wc * 64u) * 4u);
#pragma unroll
  for (int a = 0; a < 4; ++a) {
    f2 y[4][2];
#pragma unroll
    for (int b = 0; b < 4; ++b) {
      y[b][0] = gelu2(f2{acc[a][b][0], acc[a][b][1]});
      y[b][1] = gelu2(f2{acc[a][b][2], acc[a][b][3]});
    }
#pragma unroll
    for (int r = 0; r < 4; ++r) {
      const f4 o = f4{y[0][r >> 1][r & 1], y[1][r >> 1][r & 1], y[2][r >> 1][r & 1], y[3][r >> 1][r & 1]};
      st128(__builtin_bit_cast(u4, o), rs_o, vo + so + (a * 16 + r) * (NN * 4));
    }
  }
  if (hasx) {
    const int sx = (int)((15u * 128u + ti * 16u) * (NN * 4) + (tj * 128u + (unsigned)wc * 64u) * 4u) + wr * 8;
    f2 y[2][2];
#pragma unroll
    for (int b = 0; b < 2; ++b) {
      y[b][0] = gelu2(f2{acx[b][0], acx[b][1]});
      y[b][1] = gelu2(f2{acx[b][2], acx[b][3]});
    }
#pragma unroll
    for (int r = 0; r < 4; ++r) {
      const f2 o = f2{y[0][r >> 1][r & 1], y[1][r >> 1][r & 1]};
      __builtin_amdgcn_raw_buffer_store_b64(__builtin_bit_cast(u2, o), rs_o, vo + sx + r * (NN * 4), 0, AUX_SYS);
    }
  }
}

}

extern "C" void kernel_launch(int rank, int nranks, ncclComm_t comm,
                              hipStream_t stream,
                              void* const* d_in, const int* in_sizes, int n_in,
                              void* d_out, int out_size,
                              const int* in_dtypes, int out_dtype,
                              void* const* d_out_all,
                              void* const* d_ws_all, size_t ws_bytes) {
  (void)comm; (void)in_dtypes; (void)out_dtype; (void)d_out_all;
  if (nranks != NR || (unsigned)rank >= (unsigned)NR || n_in < 2 || ws_bytes < (size_t)WS_NEED ||
      in_sizes[0] != MM * KS || in_sizes[1] != KS * NN || out_size != MM * NN) {
    fprintf(stderr, "kernel_launch: unexpected problem shape\n");
    return;
  }
  static int ncu = 0;
  if (ncu == 0) {
    int dev = 0, n = 0;
    HIP_CHECK(hipGetDevice(&dev));
    HIP_CHECK(hipDeviceGetAttribute(&n, hipDeviceAttributeMultiprocessorCount, dev));
    ncu = n > 0 ? n : 1;
  }
  Params p;
  p.x = (const float*)d_in[0];
  p.w = (const float*)d_in[1];
  p.out = (float*)d_out;
  for (int r = 0; r < NR; ++r) p.ws[r] = (char*)d_ws_all[(rank + r) % NR];
  p.rank = rank;
  if (ncu >= NWG) {
    p.mode = 3;
    hipLaunchKernelGGL(fused_gemm_ag, dim3(NWG), dim3(NWAVE * 64), 0, stream, p);
  } else {
    p.mode = 1;
    hipLaunchKernelGGL(fused_gemm_ag, dim3(NWG), dim3(NWAVE * 64), 0, stream, p);
    p.mode = 2;
    hipLaunchKernelGGL(fused_gemm_ag, dim3(NWG), dim3(NWAVE * 64), 0, stream, p);
  }
  HIP_CHECK(hipGetLastError());
}
```

```cpp
#include <hip/hip_runtime.h>
#include <rccl/rccl.h>
#include <cstdio>
#include <cstdint>

#define HIP_CHECK(expr) do { hipError_t _e = (expr); if (_e != hipSuccess) { \
    fprintf(stderr, "HIP error %s at %s:%d\n", hipGetErrorString(_e), __FILE__, __LINE__); \
    return; } } while (0)

typedef _Float16 h8 __attribute__((ext_vector_type(8)));
typedef _Float16 h2 __attribute__((ext_vector_type(2)));
typedef float f4 __attribute__((ext_vector_type(4)));
typedef float f2 __attribute__((ext_vector_type(2)));
typedef unsigned u4 __attribute__((ext_vector_type(4)));
typedef unsigned u2 __attribute__((ext_vector_type(2)));

#define SLIDE 4
#define CONV_SHIFT 4
#define CONV_SLEEP 4
#define CONV_STAG 1
#define EARLY 1
#define HINT_POS 2
#define CONV_BASE 48
#define FETCH_AHEAD 3
#define AUX_POLL 17
#define POLL_SLEEP 2

namespace {

constexpr int NR = 8;
constexpr int MM = 2048, NN = 2048, KS = 256;
constexpr int NWG = 256;
constexpr int NPW = 16;
constexpr int NGW = NWG - NPW;
constexpr int NWAVE = 12;
constexpr int NCONV = 5;
constexpr int NPIECE = 512;
constexpr int PPW = NPIECE / NPW;

constexpr unsigned CTR_OFF  = 0;
constexpr unsigned TMO_OFF  = 32768;
constexpr unsigned TKT_OFF  = 36992;
constexpr unsigned ZCNT_OFF = 36864;
constexpr unsigned FLAG_OFF = 65536;
constexpr unsigned DBG_OFF  = 262144;
constexpr unsigned DATA_OFF = 1u << 20;
constexpr unsigned PAR_BYTES = 16u << 20;
constexpr unsigned TYPE_BYTES = 8u << 20;
constexpr unsigned SRC_BYTES = 1u << 20;
constexpr unsigned WS_NEED = DATA_OFF + 2 * PAR_BYTES;

constexpr int AUX_SYS = 17;
constexpr unsigned RSRC3 = 0x00020000u;
constexpr unsigned long long DEADLINE = 100000000ull;
constexpr unsigned CALL_MASK = 0x3FFFFFu;
constexpr unsigned EBIAS = 512;
constexpr int OOR = 0x40000000;

constexpr int L_SLOT = 17408;
constexpr int L_CTL = NR * L_SLOT;
constexpr int L_LANDED = L_CTL;
constexpr int L_CONS = L_CTL + 128;
constexpr int L_MISC = L_CTL + 256;
constexpr int L_BYTES = L_CTL + 512;
constexpr int RING = 16;
constexpr int L_RDY = RING * 4096;
constexpr int L_PROG = L_CTL;
constexpr unsigned R_END = 0xFFFFu, R_SKIP = 0xFFFEu;

struct Params {
  const float* x;
  const float* w;
  char* my;
  int rank;
  int mode;
  char* ws[NR];
  float* out;
};

__device__ __forceinline__ unsigned long long now() { return __builtin_amdgcn_s_memrealtime(); }

__device__ __forceinline__ unsigned pack2(float a, float b) {
  h2 h = {(_Float16)a, (_Float16)b};
  return __builtin_bit_cast(unsigned, h);
}
__device__ __forceinline__ float max3abs(float a, float b, float m) {
  float r;
  asm("v_max3_f32 %0, |%1|, |%2|, %3" : "=v"(r) : "v"(a), "v"(b), "v"(m));
  return r;
}

__device__ __forceinline__ unsigned piece_at(unsigned p) {
  const unsigned s = p >> 6, q = p & 63u;
  const unsigned type = (q >> 1) & 1u, half = q & 1u;
  const unsigned b = ((q >> 2) + 15u + type) & 15u;
  return (type << 8) | (s << 5) | (b * 2u + half);
}

__device__ __forceinline__ void st128(u4 v, __amdgpu_buffer_rsrc_t rs, int voff) {
  __builtin_amdgcn_raw_buffer_store_b128(v, rs, voff, 0, AUX_SYS);
}

__device__ __forceinline__ unsigned lds_ld(const char* smem, int off) {
  return __hip_atomic_load((const unsigned*)(smem + off), __ATOMIC_RELAXED, __HIP_MEMORY_SCOPE_WORKGROUP);
}
__device__ __forceinline__ unsigned long long lds_ld64(const char* smem, int off) {
  return __hip_atomic_load((const unsigned long long*)(smem + off), __ATOMIC_RELAXED, __HIP_MEMORY_SCOPE_WORKGROUP);
}
__device__ __forceinline__ void lds_st(char* smem, int off, unsigned v) {
  __hip_atomic_store((unsigned*)(smem + off), v, __ATOMIC_RELAXED, __HIP_MEMORY_SCOPE_WORKGROUP);
}
__device__ __forceinline__ void lds_st64(char* smem, int off, unsigned long long v) {
  __hip_atomic_store((unsigned long long*)(smem + off), v, __ATOMIC_RELAXED, __HIP_MEMORY_SCOPE_WORKGROUP);
}
__device__ __forceinline__ unsigned landed(unsigned s, unsigned e0, unsigned e1, unsigned e2) {
  return ((s + 1u) << 28) | (e0 << 8) | (e1 << 18) | (e0 == EBIAS && e1 == EBIAS && e2 == EBIAS ? s + 1u : 0u);
}
#define INL __attribute__((always_inline))
template <int V> struct IC { static constexpr int value = V; };
__device__ __forceinline__ unsigned min4(u4 v, unsigned m) { return min(min(m, min(v[0], v[1])), min(v[2], v[3])); }

#define STAMP(idx) do {} while (0)
#define STAMPV(idx, v) do {} while (0)

__global__ void __launch_bounds__(NWAVE * 64, 1) fused_gemm_ag(Params p) {
  __shared__ __attribute__((aligned(16))) char smem[L_BYTES];
  const int lane = threadIdx.x & 63;
  const int wave = __builtin_amdgcn_readfirstlane(threadIdx.x >> 6);
  const unsigned bid = blockIdx.x;
  const float* px = p.x;
  const float* pw = p.w;
  char* myws = p.my;
  unsigned rank = (unsigned)p.rank, mode = (unsigned)p.mode;
  const bool sendwg = bid < (unsigned)NPW;

  const unsigned cpos = sendwg ? (unsigned)wave * NPW + bid : NCONV * NPW + (unsigned)wave * NGW + (bid - NPW);
  const unsigned cpc = piece_at(cpos & (NPIECE - 1));
  f4 R[4][2];
  auto conv_issue = [&]() {
    const int q = lane >> 4, c = lane & 15;
    const bool isw = (cpc >> 8) != 0u;
    const unsigned s = (cpc >> 5) & 7u, g = cpc & 31u;
    const int sb = (int)(s << (isw ? 18 : 7)) + (int)(g << (isw ? 8 : 16));
    const int sa = isw ? NN * 4 : 16 * KS * 4;
    const int sbb = isw ? 16 * NN * 4 : 64;
    const int vo = (q << (isw ? 15 : 4)) + (c << (isw ? 4 : 10));
    asm volatile("" : "+s"(px), "+s"(pw), "+s"(myws), "+s"(rank), "+s"(mode));
    const __amdgpu_buffer_rsrc_t rs = __builtin_amdgcn_make_buffer_rsrc((void*)(isw ? pw : px), 0, MM * KS * 4, RSRC3);
#pragma unroll
    for (int a = 0; a < 4; ++a)
#pragma unroll
      for (int b = 0; b < 2; ++b)
        R[a][b] = __builtin_bit_cast(f4, __builtin_amdgcn_raw_buffer_load_b128(rs, vo, sb + a * sa + b * sbb, 0));
  };
  const bool early = sendwg && wave < EARLY;
  if (early) conv_issue();
  rank &= 7u;
  if (sendwg ? !(mode & 1u) : (wave > 4 && !(mode & 2u))) return;
  const __amdgpu_buffer_rsrc_t rs_my = __builtin_amdgcn_make_buffer_rsrc((void*)myws, 0, (int)WS_NEED, RSRC3);
  const int ctr_off = (int)(CTR_OFF + (bid * 16u + (unsigned)wave) * 4u);
  const int lane0 = lane == 0 ? 0 : OOR;
  const bool conv = (mode & 1u) && (sendwg ? wave < NCONV : cpos < (unsigned)NPIECE);
  const bool sender = sendwg && wave >= NCONV;
  unsigned cv = __builtin_amdgcn_raw_buffer_load_b32(rs_my, sender ? OOR : ctr_off, 0, AUX_SYS);
  if (wave == (sendwg ? 5 : 2)) {
    lds_st(smem, L_CTL + lane * 4, 0u);
    lds_st(smem, L_CTL + 256 + lane * 4, 0u);
  }
  if (sendwg && wave >= 6 && wave < 11) *(u4*)(smem + L_RDY + (wave - 6) * 1024 + lane * 16) = u4{0u, 0u, 0u, 0u};
  asm volatile("s_waitcnt lgkmcnt(0)\n\ts_barrier" ::: "memory");
  if (conv && !early) {
    const unsigned nsl = sendwg ? (unsigned)(wave - EARLY) * CONV_STAG : (cpos - CONV_BASE) >> CONV_SHIFT;
    for (unsigned i = 0; i < nsl; ++i) __builtin_amdgcn_s_sleep(CONV_SLEEP);
    conv_issue();
  }
  asm volatile("" : "+v"(cv));
  const unsigned call = __builtin_amdgcn_readfirstlane(cv) + 1u;
  const unsigned par = call & 1u;
  const unsigned tag = call & CALL_MASK;
  if (!sender && (mode & (!sendwg && wave >= 4 ? 2u : 1u))) __builtin_amdgcn_raw_buffer_store_b32(call, rs_my, lane0, ctr_off, AUX_SYS);
  auto data_off = [&](unsigned pc, unsigned pr) -> unsigned {
    return DATA_OFF + pr * PAR_BYTES + rank * SRC_BYTES + (pc >> 8) * TYPE_BYTES + (pc & 255u) * 4096u;
  };
  auto flag_off = [&](unsigned pc, unsigned pr) -> unsigned {
    return FLAG_OFF + pr * 32768u + rank * 128u + (pc >> 8) * 16384u + ((pc >> 1) & 15u) * 1024u + ((pc >> 5) & 7u) * 8u + (pc & 1u) * 4u;
  };
  if (conv) {
    float m = 0.f;
#pragma unroll
    for (int a = 0; a < 4; ++a)
#pragma unroll
      for (int b = 0; b < 2; ++b) {
        m = max3abs(R[a][b][0], R[a][b][1], m);
        m = max3abs(R[a][b][2], R[a][b][3], m);
      }
    int e = 0;
    const bool big = __builtin_amdgcn_ballot_w64(m >= 32768.f) != 0ull;
    const bool small = __builtin_amdgcn_ballot_w64(m >= 0.00390625f) == 0ull && __builtin_amdgcn_ballot_w64(m > 0.f) != 0ull;
    if (big || small) {
      float mf = 0.f;
#pragma unroll
      for (int a = 0; a < 4; ++a)
#pragma unroll
        for (int b = 0; b < 2; ++b)
#pragma unroll
          for (int t = 0; t < 4; ++t) {
            const float av = __builtin_fabsf(R[a][b][t]);
            mf = fmaxf(mf, av < __builtin_inff() ? av : 0.f);
          }
#pragma unroll
      for (int o = 32; o >= 1; o >>= 1) mf = fmaxf(mf, __shfl_xor(mf, o, 64));
      int ee = 0;
      if (mf > 0.f) { (void)__builtin_frexpf(mf, &ee); ee -= 10; }
      e = __builtin_amdgcn_readfirstlane(ee);
#pragma unroll
      for (int a = 0; a < 4; ++a)
#pragma unroll
        for (int b = 0; b < 2; ++b)
#pragma unroll
          for (int t = 0; t < 4; ++t) R[a][b][t] = __builtin_ldexpf(R[a][b][t], -e);
    }
    u4 o[4];
    if ((cpc >> 8) == 0u) {
#pragma unroll
      for (int a = 0; a < 4; ++a)
        o[a] = u4{pack2(R[a][0][0], R[a][0][1]), pack2(R[a][0][2], R[a][0][3]),
                  pack2(R[a][1][0], R[a][1][1]), pack2(R[a][1][2], R[a][1][3])};
    } else {
#pragma unroll
      for (int i = 0; i < 4; ++i)
        o[i] = u4{pack2(R[0][0][i], R[1][0][i]), pack2(R[2][0][i], R[3][0][i]),
                  pack2(R[0][1][i], R[1][1][i]), pack2(R[2][1][i], R[3][1][i])};
    }
    unsigned mn = 0xFFFFFFFFu;
#pragma unroll
    for (int i = 0; i < 4; ++i) mn = min4(o[i], mn);
    if (__builtin_amdgcn_ballot_w64(mn == 0u) != 0ull) {
#pragma unroll
      for (int i = 0; i < 4; ++i)
#pragma unroll
        for (int t = 0; t < 4; ++t) o[i][t] = o[i][t] == 0u ? 0x8000u : o[i][t];
    }
    const unsigned h = (tag << 10) | (unsigned)(e + (int)EBIAS);
    if (sendwg) {
#pragma unroll
      for (int i = 0; i < 4; ++i) *(u4*)(smem + wave * 4096 + i * 1024 + lane * 16) = o[i];
      asm volatile("" ::: "memory");
      lds_st64(smem, L_RDY + wave * 8, (unsigned long long)h | ((unsigned long long)cpos << 32));
      if (wave == 0) STAMP(5);
      asm volatile("" ::: "memory");
    }
    const int dof = (int)data_off(cpc, par) + lane * 16;
#pragma unroll
    for (int i = 0; i < 4; ++i) st128(o[i], rs_my, dof + i * 1024);
    __builtin_amdgcn_raw_buffer_store_b32(h, rs_my, lane0, (int)flag_off(cpc, par), AUX_SYS);
  }

  if (sendwg) {
    const unsigned long long t0 = now();
    if (sender) {
      const int dk = (int)(((unsigned)(wave - NCONV) + bid) % (unsigned)(NR - 1)) + 1;
      __builtin_amdgcn_s_setprio(3);
      const __amdgpu_buffer_rsrc_t rs_d = __builtin_amdgcn_make_buffer_rsrc((void*)p.ws[dk], 0, (int)WS_NEED, RSRC3);
      unsigned ends = 0;
      int drop = OOR;
      asm volatile("" : "+s"(drop));
      for (unsigned n = ~0u; ends < (unsigned)NCONV; ++n) {
        unsigned h, pos, spins = 0;
        for (;;) {
          asm volatile("" ::: "memory");
          const unsigned long long r = lds_ld64(smem, L_RDY + (int)n * 8);
          h = __builtin_amdgcn_readfirstlane((unsigned)r);
          pos = __builtin_amdgcn_readfirstlane((unsigned)(r >> 32));
          if ((h | (unsigned)drop) != 0u) break;
          if ((++spins & 255u) == 0u && now() - t0 > 2 * DEADLINE) return;
          __builtin_amdgcn_s_sleep(1);
        }
        asm volatile("" ::: "memory");
        if (drop) pos = 0;
        if (pos >= (unsigned)NPIECE) {
          ends += pos == R_END ? 1u : 0u;
          lds_st(smem, L_PROG + (dk - 1) * 4, n + 1u);
          continue;
        }
        u4 o[4];
#pragma unroll
        for (int i = 0; i < 4; ++i) o[i] = *(const u4*)(smem + (n & (RING - 1)) * 4096u + i * 1024 + lane * 16);
        asm volatile("" ::: "memory");
        lds_st(smem, L_PROG + (dk - 1) * 4, n + 1u);
        const unsigned pc = piece_at(pos);
        const int dof = ((int)data_off(pc, (h >> 10) & 1u) + lane * 16) | drop;
#pragma unroll
        for (int i = 0; i < 4; ++i) {
          asm volatile("s_waitcnt vmcnt(%0)" :: "n"(SLIDE) : "memory");
          st128(o[i], rs_d, dof + i * 1024);
          if (i == HINT_POS) __builtin_amdgcn_raw_buffer_store_b32(h, rs_d, lane0 | drop, (int)flag_off(pc, (h >> 10) & 1u), AUX_SYS);
        }
        drop = 0;
      }
      return;
    }
    for (unsigned n = (unsigned)wave + NCONV; ; n += NCONV) {
      unsigned spins = 0;
      for (;;) {
        asm volatile("" ::: "memory");
        const unsigned pr = lds_ld(smem, L_PROG + (lane < NR - 1 ? lane : 0) * 4);
        if (__builtin_amdgcn_ballot_w64(pr + FETCH_AHEAD < n) == 0ull) break;
        if ((++spins & 255u) == 0u && now() - t0 > 2 * DEADLINE) return;
        __builtin_amdgcn_s_sleep(2);
      }
      const int raw = __builtin_amdgcn_raw_ptr_buffer_atomic_add_i32(1, rs_my, lane0, (int)TKT_OFF, 0);
      const unsigned pos = ((unsigned)__builtin_amdgcn_readfirstlane(raw) & (NPIECE - 1)) + NCONV * NPW;
      if (pos >= (unsigned)NPIECE) {
        lds_st64(smem, L_RDY + (int)(n + NCONV) * 8, 1ull | ((unsigned long long)R_SKIP << 32));
        lds_st64(smem, L_RDY + (int)n * 8, 1ull | ((unsigned long long)R_END << 32));
        break;
      }
      const unsigned pc = piece_at(pos);
      for (;;) {
        u4 o[4];
#pragma unroll
        for (int i = 0; i < 4; ++i)
          o[i] = __builtin_amdgcn_raw_buffer_load_b128(rs_my, lane * 16 + i * 1024, (int)data_off(pc, par), AUX_SYS);
        const unsigned h = __builtin_amdgcn_raw_buffer_load_b32(rs_my, 0, (int)flag_off(pc, par), AUX_SYS);
        unsigned mn = (h >> 10) == tag ? 0xFFFFFFFFu : 0u;
        if (pos == (unsigned)NPIECE - 1u) {
          const unsigned z = __builtin_amdgcn_raw_buffer_load_b32(rs_my, 0, (int)ZCNT_OFF, AUX_SYS);
          if ((int)(z - call * (unsigned)NGW) < 0) mn = 0u;
        }
#pragma unroll
        for (int i = 0; i < 4; ++i) mn = min4(o[i], mn);
        if (__builtin_amdgcn_ballot_w64(mn == 0u) == 0ull) {
#pragma unroll
          for (int i = 0; i < 4; ++i) *(u4*)(smem + (n & (RING - 1)) * 4096u + i * 1024 + lane * 16) = o[i];
          asm volatile("" ::: "memory");
          lds_st64(smem, L_RDY + (int)n * 8, (unsigned long long)h | ((unsigned long long)pos << 32));
          break;
        }
        if ((++spins & 63u) == 0u && now() - t0 > DEADLINE) return;
        __builtin_amdgcn_s_sleep(4);
      }
    }
    if (wave == 0) { STAMP(8); STAMPV(0, ts00); STAMPV(6, ts0); }
    return;
  }

  const unsigned gi = (bid >> 3) - (unsigned)(NPW / 8);
  const unsigned ti = gi >> 1;
  const unsigned tj = ((bid - ti) & 7u) + (gi & 1u) * 8u;
  const bool hasx = ti < 8u;
  const int wk = wave;
  const int wr = (wk >> 1) & 1, wc = wk & 1;
  float* outp = p.out;
  asm volatile("" : "+s"(outp));
  f4 acc[4][4], acx[2];
  int odrop = 0;

  if (wave >= 4) {
    const unsigned k = (unsigned)(wave - 4);
    const unsigned src = (rank + k) & 7u;
    const unsigned ltype = (lane >> 1) & 1u;
    const int fo = (int)(FLAG_OFF + par * 32768u + src * 128u +
                         (lane == 4 && hasx ? 15u * 1024u + (ti >> 2) * 4u : ltype * 16384u + (ltype ? tj : ti) * 1024u + (lane & 1u) * 4u));
    const unsigned dx = DATA_OFF + par * PAR_BYTES + src * SRC_BYTES + ti * 8192u;
    const unsigned de = DATA_OFF + par * PAR_BYTES + src * SRC_BYTES + 15u * 8192u + ti * 1024u;
    const unsigned dw = DATA_OFF + par * PAR_BYTES + TYPE_BYTES + src * SRC_BYTES + tj * 8192u;
    char* const slot = smem + k * L_SLOT + lane * 16;
    const __amdgpu_buffer_rsrc_t rs_z = __builtin_amdgcn_make_buffer_rsrc(
        (void*)(myws + DATA_OFF + (par ^ 1u) * PAR_BYTES), 0, (int)PAR_BYTES, RSRC3);
    constexpr int ZPS = (int)((PAR_BYTES / 1024 + NGW * 8 - 1) / (NGW * 8));
    const bool zer = k == 0u && (mode & 1u);
    const unsigned long long t0 = now();
    unsigned retries = 0;
    bool dead = false;
    for (unsigned s = 0; s < 8 && !dead; ++s) {
      unsigned got = (mode & 2u) ? 0u : 3u;
      unsigned spins = 0;
      unsigned want = 0;
      unsigned f = 0;
      while (got != 3u) {
        asm volatile("" ::: "memory");
        const unsigned fnew = __builtin_amdgcn_raw_buffer_load_b32(rs_my, f ? OOR : fo, (int)(s * 8u), AUX_POLL);
        if (want) {
          u4 a[8], b[8], ae = u4{1u, 1u, 1u, 1u};
          if (want & 1u) {
#pragma unroll
            for (int i = 0; i < 8; ++i)
              a[i] = __builtin_amdgcn_raw_buffer_load_b128(rs_my, lane * 16 + (i & 3) * 1024, (int)(dx + s * 131072u + (i >> 2) * 4096u), AUX_SYS);
            if (hasx) ae = __builtin_amdgcn_raw_buffer_load_b128(rs_my, lane * 16, (int)(de + s * 131072u), AUX_SYS);
          }
          if (want & 2u) {
#pragma unroll
            for (int i = 0; i < 8; ++i)
              b[i] = __builtin_amdgcn_raw_buffer_load_b128(rs_my, lane * 16 + (i & 3) * 1024, (int)(dw + s * 131072u + (i >> 2) * 4096u), AUX_SYS);
          }
          if (s > 0) {
            unsigned sp2 = 0;
            for (;;) {
              asm volatile("" ::: "memory");
              const unsigned cv = lds_ld(smem, L_CONS + (int)k * 16 + (lane & 3) * 4);
              if (__builtin_amdgcn_ballot_w64(cv < s) == 0ull) break;
              if ((++sp2 & 255u) == 0u && (lds_ld(smem, L_MISC) != 0u || now() - t0 > DEADLINE)) { dead = true; break; }
              __builtin_amdgcn_s_sleep(1);
            }
            if (dead) break;
          }
          asm volatile("" ::: "memory");
          if (want & 1u) {
            const unsigned e0 = __builtin_amdgcn_readlane(f, 0) & 1023u, e1 = __builtin_amdgcn_readlane(f, 1) & 1023u;
            const unsigned e2 = __builtin_amdgcn_readlane(f, 4) & 1023u;
            lds_st(smem, L_LANDED + (int)k * 16 + 8, e2);
            unsigned mn = 0xFFFFFFFFu;
#pragma unroll
            for (int i = 0; i < 8; ++i) { *(u4*)(slot + i * 1024) = a[i]; mn = min4(a[i], mn); }
            *(u4*)(slot + 16384) = ae;
            mn = min4(ae, mn);
            asm volatile("" ::: "memory");
            if (__builtin_amdgcn_ballot_w64(mn == 0u) == 0ull) {
              lds_st(smem, L_LANDED + (int)k * 16, landed(s, e0, e1, e2));
              got |= 1u;
            } else ++retries;
          }
          if (want & 2u) {
            const unsigned e0 = __builtin_amdgcn_readlane(f, 2) & 1023u, e1 = __builtin_amdgcn_readlane(f, 3) & 1023u;
            unsigned mn = 0xFFFFFFFFu;
#pragma unroll
            for (int i = 0; i < 8; ++i) { *(u4*)(slot + 8192 + i * 1024) = b[i]; mn = min4(b[i], mn); }
            asm volatile("" ::: "memory");
            if (__builtin_amdgcn_ballot_w64(mn == 0u) == 0ull) {
              lds_st(smem, L_LANDED + (int)k * 16 + 4, landed(s, e0, e1, EBIAS));
              got |= 2u;
            } else ++retries;
          }
          if ((retries & 1023u) == 1023u && now() - t0 > DEADLINE) { dead = true; break; }
        }
        f = (fnew >> 10) == tag && fnew ? fnew : f;
        const unsigned rdy = (unsigned)__builtin_amdgcn_ballot_w64(f != 0u);
        want = (((rdy & 19u) == 19u ? 1u : 0u) | ((rdy & 12u) == 12u ? 2u : 0u)) & ~got;
        if (want == 0u && got != 3u) {
          if ((++spins & 63u) == 0u) {
            const unsigned tm = __builtin_amdgcn_readfirstlane(__builtin_amdgcn_raw_buffer_load_b32(rs_my, (int)TMO_OFF, 0, AUX_SYS));
            if (tm == call || now() - t0 > DEADLINE) { dead = true; break; }
          }
          __builtin_amdgcn_s_sleep(POLL_SLEEP);
        }
      }
      if (zer) {
#pragma unroll
        for (int i = 0; i < ZPS; ++i)
          st128(u4{0u, 0u, 0u, 0u}, rs_z, (int)((bid - NPW + (s * ZPS + i) * NGW) * 1024u) + lane * 16);
      }
    }
    if (zer && !dead) {
      asm volatile("s_waitcnt vmcnt(0)" ::: "memory");
      (void)__builtin_amdgcn_raw_ptr_buffer_atomic_add_i32(1, rs_my, lane0, (int)ZCNT_OFF, 0);
    }
    if (!(mode & 2u)) return;
    if (dead) {
      __builtin_amdgcn_raw_buffer_store_b32(call, rs_my, lane0, (int)TMO_OFF, AUX_SYS);
      lds_st(smem, L_MISC, 1u);
    }
    STAMP(8 + k);
    STAMPV(16 + k, retries);
    STAMPV(24 + k, tseen);
    if (k != 0u) return;
    odrop = OOR;
#pragma unroll
    for (int a = 0; a < 4; ++a)
#pragma unroll
      for (int b = 0; b < 4; ++b) acc[a][b] = f4{0.f, 0.f, 0.f, 0.f};
    acx[0] = acx[1] = f4{0.f, 0.f, 0.f, 0.f};
  } else {
  if (wk == 0) STAMPV(4, ts00);
  if (wk == 0) STAMP(2);
  if (!(mode & 2u)) return;
#pragma unroll
  for (int a = 0; a < 4; ++a)
#pragma unroll
    for (int b = 0; b < 4; ++b) acc[a][b] = f4{0.f, 0.f, 0.f, 0.f};
  acx[0] = acx[1] = f4{0.f, 0.f, 0.f, 0.f};

  unsigned done = 0;
  unsigned nd = 0;
  const char* const abase = smem + wr * 4096 + lane * 16;
  const char* const bbase = smem + 8192 + wc * 4096 + lane * 16;
  const char* const ebase = smem + 16384 + lane * 16;
  const int cons = L_CONS + (lane & 7) * 16 + wk * 4;
  const unsigned long long t0 = now();
  struct Frag { h8 b[4], a[2], e; };
  h8 af[2];
  Frag f0, f1;
  unsigned lx, lw;
  auto peek = [&]() INL {
    asm volatile("" ::: "memory");
    const unsigned long long l = lds_ld64(smem, L_LANDED + (lane & 7) * 16);
    lx = (unsigned)l; lw = (unsigned)(l >> 32);
    asm volatile("" ::: "memory");
  };
  auto take = [&](int k) INL {
    asm volatile("" ::: "memory");
    done += (lane & 7) == k ? 1u : 0u;
    lds_st(smem, cons, done);
    asm volatile("" ::: "memory");
  };
  auto read = [&](Frag& fn, int k) INL {
    asm volatile("" ::: "memory");
#pragma unroll
    for (int i = 0; i < 4; ++i) fn.b[i] = *(const h8*)(bbase + k * L_SLOT + i * 1024);
#pragma unroll
    for (int i = 0; i < 2; ++i) af[i] = *(const h8*)(abase + k * L_SLOT + i * 1024);
#pragma unroll
    for (int i = 0; i < 2; ++i) fn.a[i] = *(const h8*)(abase + k * L_SLOT + 2048 + i * 1024);
    fn.e = *(const h8*)(ebase + k * L_SLOT);
    asm volatile("" ::: "memory");
  };
  enum { W_GO = 0, W_END, W_SLOW, W_DEAD };
  auto wait_and_read = [&](Frag& fn) INL -> int {
    if (nd == 64u) return W_END;
    unsigned rdy, spins = 0;
    for (;;) {
      peek();
      rdy = (unsigned)__builtin_amdgcn_ballot_w64(min(lx & 255u, lw & 255u) > done) & 255u;
      if (rdy) break;
      if (__builtin_amdgcn_ballot_w64(min(lx, lw) >> 28 > done) != 0ull) return W_SLOW;
      if ((++spins & 63u) == 0u && (lds_ld(smem, L_MISC) != 0u || now() - t0 > 2 * DEADLINE)) return W_DEAD;
      __builtin_amdgcn_s_sleep(1);
    }
    const int k = __builtin_ctz(rdy);
    read(fn, k);
    take(k);
    peek();
    ++nd;
    return W_GO;
  };
#define SB() __builtin_amdgcn_sched_barrier(0)
#define MF(A, B) acc[A][B] = __builtin_amdgcn_mfma_f32_16x16x32_f16((A) < 2 ? af[(A) & 1] : f.a[(A) & 1], f.b[B], acc[A][B], 0, 0, 0); SB()
  auto body = [&](auto xs, Frag& f, Frag& fn) INL -> int {
    constexpr int XS = decltype(xs)::value;
    SB();
    MF(0, 0); MF(0, 1);
    const unsigned rdy = (unsigned)__builtin_amdgcn_ballot_w64(min(lx & 255u, lw & 255u) > done) | 256u;
    SB();
    MF(0, 2);
    const int k = __builtin_ctz(rdy);
    const int ko = (k & 7) * L_SLOT;
    const char* const bp = bbase + ko;
    const char* const ap = abase + ko;
    SB();
    MF(0, 3);
    asm volatile("" ::: "memory");
    fn.b[0] = *(const h8*)(bp); SB();
    MF(1, 0);
    fn.b[1] = *(const h8*)(bp + 1024); SB();
    MF(1, 1);
    fn.b[2] = *(const h8*)(bp + 2048); SB();
    MF(1, 2);
    fn.b[3] = *(const h8*)(bp + 3072); SB();
    MF(1, 3);
    af[0] = *(const h8*)(ap); SB();
    MF(2, 0);
    af[1] = *(const h8*)(ap + 1024); SB();
    MF(2, 1);
    fn.a[0] = *(const h8*)(ap + 2048); SB();
    MF(2, 2);
    fn.a[1] = *(const h8*)(ap + 3072); SB();
    MF(2, 3);
    if (XS) fn.e = *(const h8*)(ebase + ko);
    SB();
    MF(3, 0);
    take(k); SB();
    MF(3, 1);
    peek(); SB();
    MF(3, 2); MF(3, 3);
    if (XS) {
      acx[0] = __builtin_amdgcn_mfma_f32_16x16x32_f16(f.e, f.b[XS == 1 ? 0 : 2], acx[0], 0, 0, 0);
      acx[1] = __builtin_amdgcn_mfma_f32_16x16x32_f16(f.e, f.b[XS == 1 ? 1 : 3], acx[1], 0, 0, 0);
      SB();
    }
    if (__builtin_expect(k == 8, 0)) return wait_and_read(fn);
    ++nd;
    return W_GO;
  };
  auto fast = [&](auto xs) INL -> int {
    int st = wait_and_read(f0);
    while (st == W_GO) {
      st = body(xs, f0, f1);
      if (st != W_GO) break;
      st = body(xs, f1, f0);
    }
    return st;
  };
  int st = !hasx ? fast(IC<0>{}) : wr == 0 ? fast(IC<1>{}) : fast(IC<2>{});
  for (; st == W_SLOW && nd < 64u; ++nd) {
    unsigned rdy, spins = 0;
    for (;;) {
      peek();
      rdy = (unsigned)__builtin_amdgcn_ballot_w64(min(lx, lw) >> 28 > done) & 255u;
      if (rdy) break;
      if ((++spins & 63u) == 0u && (lds_ld(smem, L_MISC) != 0u || now() - t0 > 2 * DEADLINE)) { st = W_DEAD; rdy = 1u; break; }
      __builtin_amdgcn_s_sleep(1);
    }
    const int k = __builtin_ctz(rdy);
    const unsigned se = __builtin_amdgcn_readfirstlane(lds_ld(smem, L_LANDED + k * 16 + 8));
    read(f0, k);
    take(k);
    const unsigned sx = __builtin_amdgcn_readlane(lx, k), sw = __builtin_amdgcn_readlane(lw, k);
    const int ew = (int)((sw >> (8 + 10 * wc)) & 1023u) - 2 * (int)EBIAS;
    const int e = (int)((sx >> (8 + 10 * wr)) & 1023u) + ew;
    const int e2 = (int)se + ew;
#pragma unroll
    for (int a = 0; a < 4; ++a)
#pragma unroll
      for (int b = 0; b < 4; ++b) {
        const f4 t = __builtin_amdgcn_mfma_f32_16x16x32_f16(a < 2 ? af[a & 1] : f0.a[a & 1], f0.b[b], f4{0.f, 0.f, 0.f, 0.f}, 0, 0, 0);
#pragma unroll
        for (int r = 0; r < 4; ++r) acc[a][b][r] += __builtin_ldexpf(t[r], e);
        SB();
      }
    const f4 u0 = __builtin_amdgcn_mfma_f32_16x16x32_f16(f0.e, wr ? f0.b[2] : f0.b[0], f4{0.f, 0.f, 0.f, 0.f}, 0, 0, 0);
    const f4 u1 = __builtin_amdgcn_mfma_f32_16x16x32_f16(f0.e, wr ? f0.b[3] : f0.b[1], f4{0.f, 0.f, 0.f, 0.f}, 0, 0, 0);
#pragma unroll
    for (int r = 0; r < 4; ++r) { acx[0][r] += __builtin_ldexpf(u0[r], e2); acx[1][r] += __builtin_ldexpf(u1[r], e2); }
  }
  const bool dead = st == W_DEAD;
  if (wk == 0) STAMP(5);
  if (dead) {
    const float qn = __builtin_nanf("");
#pragma unroll
    for (int a = 0; a < 4; ++a)
#pragma unroll
      for (int b = 0; b < 4; ++b) acc[a][b] = f4{qn, qn, qn, qn};
    acx[0] = acx[1] = f4{qn, qn, qn, qn};
  }
  }

  const __amdgpu_buffer_rsrc_t rs_o = __builtin_amdgcn_make_buffer_rsrc((void*)outp, 0, MM * NN * 4, RSRC3);
  const float K1 = -2.0f * 0.7978845608028654f * 1.4426950408889634f;
  const float K2 = K1 * 0.044715f;
  auto gelu2 = [&](f2 v) -> f2 {
    const f2 t = v * __builtin_elementwise_fma(v * v, f2{K2, K2}, f2{K1, K1});
    const f2 d = f2{__builtin_amdgcn_exp2f(t[0]), __builtin_amdgcn_exp2f(t[1])} + f2{1.f, 1.f};
    return v * f2{__builtin_amdgcn_rcpf(d[0]), __builtin_amdgcn_rcpf(d[1])};
  };
  const int vo = (((lane >> 4) * 4) * (NN * 4) + (lane & 15) * 16) | odrop;
  const int so = (int)((ti * 128u + (unsigned)wr * 64u) * (NN * 4) + (tj * 128u + (unsigned)wc * 64u) * 4u);
#pragma unroll
  for (int a = 0; a < 4; ++a) {
    f2 y[4][2];
#pragma unroll
    for (int b = 0; b < 4; ++b) {
      y[b][0] = gelu2(f2{acc[a][b][0], acc[a][b][1]});
      y[b][1] = gelu2(f2{acc[a][b][2], acc[a][b][3]});
    }
#pragma unroll
    for (int r = 0; r < 4; ++r) {
      const f4 o = f4{y[0][r >> 1][r & 1], y[1][r >> 1][r & 1], y[2][r >> 1][r & 1], y[3][r >> 1][r & 1]};
      st128(__builtin_bit_cast(u4, o), rs_o, vo + so + (a * 16 + r) * (NN * 4));
    }
  }
  if (hasx) {
    const int sx = (int)((15u * 128u + ti * 16u) * (NN * 4) + (tj * 128u + (unsigned)wc * 64u) * 4u) + wr * 8;
    f2 y[2][2];
#pragma unroll
    for (int b = 0; b < 2; ++b) {
      y[b][0] = gelu2(f2{acx[b][0], acx[b][1]});
      y[b][1] = gelu2(f2{acx[b][2], acx[b][3]});
    }
#pragma unroll
    for (int r = 0; r < 4; ++r) {
      const f2 o = f2{y[0][r >> 1][r & 1], y[1][r >> 1][r & 1]};
      __builtin_amdgcn_raw_buffer_store_b64(__builtin_bit_cast(u2, o), rs_o, vo + sx + r * (NN * 4), 0, AUX_SYS);
    }
  }
}

}

extern "C" void kernel_launch(int rank, int nranks, ncclComm_t comm,
                              hipStream_t stream,
                              void* const* d_in, const int* in_sizes, int n_in,
                              void* d_out, int out_size,
                              const int* in_dtypes, int out_dtype,
                              void* const* d_out_all,
                              void* const* d_ws_all, size_t ws_bytes) {
  (void)comm; (void)in_dtypes; (void)out_dtype; (void)d_out_all;
  if (nranks != NR || (unsigned)rank >= (unsigned)NR || n_in < 2 || ws_bytes < (size_t)WS_NEED ||
      in_sizes[0] != MM * KS || in_sizes[1] != KS * NN || out_size != MM * NN) {
    fprintf(stderr, "kernel_launch: unexpected problem shape\n");
    return;
  }
  static int ncu = 0;
  if (ncu == 0) {
    int dev = 0, n = 0;
    HIP_CHECK(hipGetDevice(&dev));
    HIP_CHECK(hipDeviceGetAttribute(&n, hipDeviceAttributeMultiprocessorCount, dev));
    ncu = n > 0 ? n : 1;
  }
  Params p;
  p.x = (const float*)d_in[0];
  p.w = (const float*)d_in[1];
  p.out = (float*)d_out;
  for (int r = 0; r < NR; ++r) p.ws[r] = (char*)d_ws_all[(rank + r) % NR];
  p.my = p.ws[0];
  p.rank = rank;
  if (ncu >= NWG) {
    p.mode = 3;
    hipLaunchKernelGGL(fused_gemm_ag, dim3(NWG), dim3(NWAVE * 64), 0, stream, p);
  } else {
    p.mode = 1;
    hipLaunchKernelGGL(fused_gemm_ag, dim3(NWG), dim3(NWAVE * 64), 0, stream, p);
    p.mode = 2;
    hipLaunchKernelGGL(fused_gemm_ag, dim3(NWG), dim3(NWAVE * 64), 0, stream, p);
  }
  HIP_CHECK(hipGetLastError());
}
```

```cpp
#include <hip/hip_runtime.h>
#include <rccl/rccl.h>
#include <cstdio>
#include <cstdint>

#define HIP_CHECK(expr) do { hipError_t _e = (expr); if (_e != hipSuccess) { \
    fprintf(stderr, "HIP error %s at %s:%d\n", hipGetErrorString(_e), __FILE__, __LINE__); \
    return; } } while (0)

typedef _Float16 h8 __attribute__((ext_vector_type(8)));
typedef _Float16 h2 __attribute__((ext_vector_type(2)));
typedef float f4 __attribute__((ext_vector_type(4)));
typedef float f2 __attribute__((ext_vector_type(2)));
typedef unsigned u4 __attribute__((ext_vector_type(4)));
typedef unsigned u2 __attribute__((ext_vector_type(2)));
typedef unsigned short us2 __attribute__((ext_vector_type(2)));

#define SLIDE 4
#define CONV_SHIFT 4
#define CONV_SLEEP 4
#define CONV_STAG 1
#define EARLY 1
#define HINT_POS 2
#define CONV_BASE 48
#define FETCH_AHEAD 3
#define AUX_POLL 17
#define POLL_SLEEP 2
#define ZFLUSH 5
#define ZSTART 4
#define RAW_HEAD 32
#define RAW_TAIL 48

namespace {

constexpr int NR = 8;
constexpr int MM = 2048, NN = 2048, KS = 256;
constexpr int NWG = 256;
constexpr int NPW = 16;
constexpr int NGW = NWG - NPW;
constexpr int NWAVE = 12;
constexpr int NCONV = 5;
constexpr int NPIECE = 512;
constexpr int PPW = NPIECE / NPW;

constexpr unsigned CTR_OFF  = 0;
constexpr unsigned TMO_OFF  = 32768;
constexpr unsigned TKT_OFF  = 36992;
constexpr unsigned ZCNT_OFF = 36864;
constexpr unsigned DBG_OFF  = 262144;
constexpr unsigned DATA_OFF = 1u << 20;
constexpr unsigned PS = 4224;
constexpr unsigned SLICE_BYTES = 32 * PS;
constexpr unsigned SRC_BYTES = 8 * SLICE_BYTES;
constexpr unsigned TYPE_BYTES = 8 * SRC_BYTES;
constexpr unsigned PAR_BYTES = 2 * TYPE_BYTES;
constexpr unsigned WS_NEED = DATA_OFF + 2 * PAR_BYTES;

constexpr int AUX_SYS = 17;
constexpr unsigned RSRC3 = 0x00020000u;
constexpr unsigned long long DEADLINE = 100000000ull;
constexpr unsigned CALL_MASK = 0x7FFFu;
constexpr unsigned H_PACKED = 1u << 15, H_LONG = 1u << 16;
constexpr unsigned EBIAS = 512;
constexpr int OOR = 0x40000000;

constexpr int L_SLOT = 17408;
constexpr int L_CTL = NR * L_SLOT;
constexpr int L_LANDED = L_CTL;
constexpr int L_CONS = L_CTL + 128;
constexpr int L_MISC = L_CTL + 256;
constexpr int L_HDR = L_CTL + 512;
constexpr int L_BYTES = L_CTL + 768;
constexpr int RING = 16;
constexpr int L_RDY = RING * (int)PS;
constexpr int L_PROG = L_CTL;
constexpr unsigned R_END = 0xFFFFu, R_SKIP = 0xFFFEu;

struct Params {
  const float* x;
  const float* w;
  char* my;
  int rank;
  int mode;
  char* ws[NR];
  float* out;
};

__device__ __forceinline__ unsigned long long now() { return __builtin_amdgcn_s_memrealtime(); }

__device__ __forceinline__ unsigned pack2(float a, float b) {
  h2 h = {(_Float16)a, (_Float16)b};
  return __builtin_bit_cast(unsigned, h);
}
__device__ __forceinline__ float max3abs(float a, float b, float m) {
  float r;
  asm("v_max3_f32 %0, |%1|, |%2|, %3" : "=v"(r) : "v"(a), "v"(b), "v"(m));
  return r;
}

__device__ __forceinline__ unsigned piece_at(unsigned p) {
  const unsigned s = p >> 6, q = p & 63u;
  const unsigned type = (q >> 1) & 1u, half = q & 1u;
  const unsigned b = ((q >> 2) + 15u + type) & 15u;
  return (type << 8) | (s << 5) | (b * 2u + half);
}

__device__ __forceinline__ void st128(u4 v, __amdgpu_buffer_rsrc_t rs, int voff) {
  __builtin_amdgcn_raw_buffer_store_b128(v, rs, voff, 0, AUX_SYS);
}

__device__ __forceinline__ unsigned lds_ld(const char* smem, int off) {
  return __hip_atomic_load((const unsigned*)(smem + off), __ATOMIC_RELAXED, __HIP_MEMORY_SCOPE_WORKGROUP);
}
__device__ __forceinline__ unsigned long long lds_ld64(const char* smem, int off) {
  return __hip_atomic_load((const unsigned long long*)(smem + off), __ATOMIC_RELAXED, __HIP_MEMORY_SCOPE_WORKGROUP);
}
__device__ __forceinline__ void lds_st(char* smem, int off, unsigned v) {
  __hip_atomic_store((unsigned*)(smem + off), v, __ATOMIC_RELAXED, __HIP_MEMORY_SCOPE_WORKGROUP);
}
__device__ __forceinline__ void lds_st64(char* smem, int off, unsigned long long v) {
  __hip_atomic_store((unsigned long long*)(smem + off), v, __ATOMIC_RELAXED, __HIP_MEMORY_SCOPE_WORKGROUP);
}
__device__ __forceinline__ void lds_xor(char* smem, int off, unsigned v) {
  (void)__hip_atomic_fetch_xor((unsigned*)(smem + off), v, __ATOMIC_RELAXED, __HIP_MEMORY_SCOPE_WORKGROUP);
}
__device__ __forceinline__ unsigned landed(unsigned s, unsigned e0, unsigned e1, unsigned e2) {
  return ((s + 1u) << 28) | (e0 << 8) | (e1 << 18) | (e0 == EBIAS && e1 == EBIAS && e2 == EBIAS ? s + 1u : 0u);
}
#define INL __attribute__((always_inline))
template <int V> struct IC { static constexpr int value = V; };
__device__ __forceinline__ unsigned min4(u4 v, unsigned m) { return min(min(m, min(v[0], v[1])), min(v[2], v[3])); }

#define STAMP(idx) do {} while (0)
#define STAMPV(idx, v) do {} while (0)

__global__ void __launch_bounds__(NWAVE * 64, 1) fused_gemm_ag(Params p) {
  __shared__ __attribute__((aligned(16))) char smem[L_BYTES];
  const int lane = threadIdx.x & 63;
  const int wave = __builtin_amdgcn_readfirstlane(threadIdx.x >> 6);
  const unsigned bid = blockIdx.x;
  const float* px = p.x;
  const float* pw = p.w;
  char* myws = p.my;
  unsigned rank = (unsigned)p.rank, mode = (unsigned)p.mode;
  const bool sendwg = bid < (unsigned)NPW;

  const unsigned cpos = sendwg ? (unsigned)wave * NPW + bid : NCONV * NPW + (unsigned)wave * NGW + (bid - NPW);
  const unsigned cpc = piece_at(cpos & (NPIECE - 1));
  f4 R[4][2];
  auto conv_issue = [&]() {
    const int q = lane >> 4, c = lane & 15;
    const bool isw = (cpc >> 8) != 0u;
    const unsigned s = (cpc >> 5) & 7u, g = cpc & 31u;
    const int sb = (int)(s << (isw ? 18 : 7)) + (int)(g << (isw ? 8 : 16));
    const int sa = isw ? NN * 4 : 16 * KS * 4;
    const int sbb = isw ? 16 * NN * 4 : 64;
    const int vo = (q << (isw ? 15 : 4)) + (c << (isw ? 4 : 10));
    asm volatile("" : "+s"(px), "+s"(pw), "+s"(myws), "+s"(rank), "+s"(mode));
    const __amdgpu_buffer_rsrc_t rs = __builtin_amdgcn_make_buffer_rsrc((void*)(isw ? pw : px), 0, MM * KS * 4, RSRC3);
#pragma unroll
    for (int a = 0; a < 4; ++a)
#pragma unroll
      for (int b = 0; b < 2; ++b)
        R[a][b] = __builtin_bit_cast(f4, __builtin_amdgcn_raw_buffer_load_b128(rs, vo, sb + a * sa + b * sbb, 0));
  };
  const bool early = sendwg && wave < EARLY;
  if (early) conv_issue();
  rank &= 7u;
  if (sendwg && !(mode & 1u)) return;
  const __amdgpu_buffer_rsrc_t rs_my = __builtin_amdgcn_make_buffer_rsrc((void*)myws, 0, (int)WS_NEED, RSRC3);
  const int ctr_off = (int)(CTR_OFF + (bid * 16u + (unsigned)wave) * 4u);
  const int lane0 = lane == 0 ? 0 : OOR;
  const bool conv = (mode & 1u) && (sendwg ? wave < NCONV : cpos < (unsigned)NPIECE);
  const bool sender = sendwg && wave >= NCONV;
  unsigned cv = __builtin_amdgcn_raw_buffer_load_b32(rs_my, sender ? OOR : ctr_off, 0, AUX_SYS);
  if (wave == (sendwg ? 5 : 2)) {
    lds_st(smem, L_CTL + lane * 4, 0u);
    lds_st(smem, L_CTL + 256 + lane * 4, 0u);
  }
  if (sendwg && wave >= 6 && wave < 11) *(u4*)(smem + L_RDY + (wave - 6) * 1024 + lane * 16) = u4{0u, 0u, 0u, 0u};
  asm volatile("s_waitcnt lgkmcnt(0)\n\ts_barrier" ::: "memory");
  if (conv && !early) {
    const unsigned nsl = sendwg ? (unsigned)(wave - EARLY) * CONV_STAG : (cpos - CONV_BASE) >> CONV_SHIFT;
    for (unsigned i = 0; i < nsl; ++i) __builtin_amdgcn_s_sleep(CONV_SLEEP);
    conv_issue();
  }
  asm volatile("" : "+v"(cv));
  const unsigned call = __builtin_amdgcn_readfirstlane(cv) + 1u;
  const unsigned par = call & 1u;
  const unsigned tag = call & CALL_MASK;
  if (!sender && (mode & (!sendwg && wave >= 4 ? 2u : 1u))) __builtin_amdgcn_raw_buffer_store_b32(call, rs_my, lane0, ctr_off, AUX_SYS);
  auto slot_off = [&](unsigned pc, unsigned pr) -> unsigned {
    return DATA_OFF + pr * PAR_BYTES + rank * SRC_BYTES + (pc >> 8) * TYPE_BYTES + (pc & 255u) * PS;
  };
  if (conv) {
    float m = 0.f;
#pragma unroll
    for (int a = 0; a < 4; ++a)
#pragma unroll
      for (int b = 0; b < 2; ++b) {
        m = max3abs(R[a][b][0], R[a][b][1], m);
        m = max3abs(R[a][b][2], R[a][b][3], m);
      }
    int e = 0;
    const bool big = __builtin_amdgcn_ballot_w64(m >= 32768.f) != 0ull;
    const bool small = __builtin_amdgcn_ballot_w64(m >= 0.00390625f) == 0ull && __builtin_amdgcn_ballot_w64(m > 0.f) != 0ull;
    if (big || small) {
      float mf = 0.f;
#pragma unroll
      for (int a = 0; a < 4; ++a)
#pragma unroll
        for (int b = 0; b < 2; ++b)
#pragma unroll
          for (int t = 0; t < 4; ++t) {
            const float av = __builtin_fabsf(R[a][b][t]);
            mf = fmaxf(mf, av < __builtin_inff() ? av : 0.f);
          }
#pragma unroll
      for (int o = 32; o >= 1; o >>= 1) mf = fmaxf(mf, __shfl_xor(mf, o, 64));
      int ee = 0;
      if (mf > 0.f) { (void)__builtin_frexpf(mf, &ee); ee -= 10; }
      e = __builtin_amdgcn_readfirstlane(ee);
#pragma unroll
      for (int a = 0; a < 4; ++a)
#pragma unroll
        for (int b = 0; b < 2; ++b)
#pragma unroll
          for (int t = 0; t < 4; ++t) R[a][b][t] = __builtin_ldexpf(R[a][b][t], -e);
    }
    u4 o[4];
    if ((cpc >> 8) == 0u) {
#pragma unroll
      for (int a = 0; a < 4; ++a)
        o[a] = u4{pack2(R[a][0][0], R[a][0][1]), pack2(R[a][0][2], R[a][0][3]),
                  pack2(R[a][1][0], R[a][1][1]), pack2(R[a][1][2], R[a][1][3])};
    } else {
#pragma unroll
      for (int i = 0; i < 4; ++i)
        o[i] = u4{pack2(R[0][0][i], R[1][0][i]), pack2(R[2][0][i], R[3][0][i]),
                  pack2(R[0][1][i], R[1][1][i]), pack2(R[2][1][i], R[3][1][i])};
    }
    unsigned mn = 0xFFFFFFFFu;
#pragma unroll
    for (int i = 0; i < 4; ++i) mn = min4(o[i], mn);
    if (__builtin_amdgcn_ballot_w64(mn == 0u) != 0ull) {
#pragma unroll
      for (int i = 0; i < 4; ++i)
#pragma unroll
        for (int t = 0; t < 4; ++t) o[i][t] = o[i][t] == 0u ? 0x8000u : o[i][t];
    }
    unsigned h = (tag << 17) | (unsigned)(e + (int)EBIAS);
    char* const hdr = smem + (sendwg ? wave * (int)PS : L_HDR + wave * 128);
    if (cpos >= (unsigned)RAW_HEAD && cpos < (unsigned)(NPIECE - RAW_TAIL) && (cpc & 0x11Eu) != 0x1Eu) {
      *(unsigned*)(hdr + (lane & 31) * 4) = 0x80008000u;
      const unsigned eh = (pack2(__builtin_ldexpf(m, -e), 0.f) >> 10) & 31u;
      unsigned em = 0;
#pragma unroll
      for (unsigned bit = 16; bit; bit >>= 1) em |= __builtin_amdgcn_ballot_w64(eh >= (em | bit)) != 0ull ? bit : 0u;
      const float top = __builtin_bit_cast(float, (em + 112u) << 23), low = __builtin_bit_cast(float, (em + 105u) << 23);
      int nt = 0, nl = 0;
#pragma unroll
      for (int a = 0; a < 4; ++a)
#pragma unroll
        for (int t = 0; t < 4; ++t) {
          nt += __builtin_popcountll(__builtin_amdgcn_ballot_w64(__builtin_fabsf(R[a][0][t]) >= top));
          nl += __builtin_popcountll(__builtin_amdgcn_ballot_w64(__builtin_fabsf(R[a][0][t]) < low));
        }
      const unsigned base = (unsigned)max((int)em - (2 * nt < nl ? 8 : 7), 0);
      const unsigned bb = base * 0x04000400u;
      unsigned T[2][8], M[2] = {0u, 0u};
      mn = 0xFFFFFFFFu;
#pragma unroll
      for (int g = 0; g < 2; ++g) {
#pragma unroll
        for (int j = 0; j < 8; ++j) {
          const unsigned d = o[2 * g + (j >> 2)][j & 3];
          const unsigned v = __builtin_bit_cast(unsigned, __builtin_bit_cast(us2, d) - __builtin_bit_cast(us2, bb)) & 0x1FFF1FFFu;
          const unsigned xm = ((v + bb) ^ d) & 0x60006000u;
          M[g] |= j < 7 ? xm >> (13 - 2 * j) : xm << 1;
          T[g][j] = v | (d & 0x80008000u);
        }
#pragma unroll
        for (int j = 0; j < 6; ++j) T[g][j] |= (T[g][7] << (13 - 2 * j)) & 0x60006000u;
        T[g][6] |= ((T[g][7] << 1) & 0x20002000u) | ((T[g][7] >> 1) & 0x40004000u);
#pragma unroll
        for (int j = 0; j < 7; ++j) mn = min(mn, T[g][j]);
      }
      unsigned cnt = __builtin_amdgcn_ballot_w64(mn == 0u) != 0ull ? 64u : 0u;
      asm volatile("" ::: "memory");
#pragma unroll
      for (int g = 0; g < 2; ++g) {
        unsigned mg = M[g];
        while (cnt <= 62u) {
          const unsigned long long bal = __builtin_amdgcn_ballot_w64(mg != 0u);
          if (bal == 0ull) break;
          const unsigned n = cnt + __builtin_amdgcn_mbcnt_hi((unsigned)(bal >> 32), __builtin_amdgcn_mbcnt_lo((unsigned)bal, 0u));
          const unsigned b = (unsigned)__builtin_ctz(mg | 0x80000000u) & 30u;
          const unsigned en = 0x8000u | (((mg >> b) & 3u) << 13) | (unsigned)(g << 11) | ((b & 8u) << 7) | (unsigned)(lane << 4) | ((b & 6u) << 1) | (b >> 4);
          if (mg != 0u && n < 62u) *(unsigned short*)(hdr + (61u - n) * 2u) = (unsigned short)en;
          mg &= ~(3u << b);
          cnt += (unsigned)__builtin_popcountll(bal);
        }
      }
      if (cnt <= 62u) {
        h |= (base << 10) | H_PACKED | (cnt > 30u ? H_LONG : 0u);
        o[0] = u4{T[0][0], T[0][1], T[0][2], T[0][3]};
        o[1] = u4{T[1][0], T[1][1], T[1][2], T[1][3]};
        o[2] = u4{T[0][4], T[0][5], T[1][4], T[1][5]};
        o[3][0] = T[0][6];
        o[3][1] = T[1][6];
      }
    }
    const bool packed = (h & H_PACKED) != 0u;
    asm volatile("" ::: "memory");
    lds_st(hdr, 124, h);
    if (sendwg) {
#pragma unroll
      for (int i = 0; i < 3; ++i) *(u4*)(hdr + 128 + i * 1024 + lane * 16) = o[i];
      if (packed) *(u2*)(hdr + 3200 + lane * 8) = u2{o[3][0], o[3][1]};
      else *(u4*)(hdr + 3200 + lane * 16) = o[3];
      asm volatile("" ::: "memory");
      lds_st64(smem, L_RDY + wave * 8, (unsigned long long)h | ((unsigned long long)cpos << 32));
      if (wave == 0) STAMP(5);
      asm volatile("" ::: "memory");
    }
    const int dof = (int)slot_off(cpc, par);
    const unsigned hv = lds_ld(hdr, (lane & 31) * 4);
#pragma unroll
    for (int i = 0; i < 3; ++i) st128(o[i], rs_my, dof + 128 + i * 1024 + lane * 16);
    if (packed) __builtin_amdgcn_raw_buffer_store_b64(u2{o[3][0], o[3][1]}, rs_my, dof + 3200 + lane * 8, 0, AUX_SYS);
    else st128(o[3], rs_my, dof + 3200 + lane * 16);
    __builtin_amdgcn_raw_buffer_store_b32(hv, rs_my, (packed ? lane < 32 : lane == 31) ? dof + lane * 4 : OOR, 0, AUX_SYS);
  }

  if (sendwg) {
    const unsigned long long t0 = now();
    if (sender) {
      const int dk = (int)(((unsigned)(wave - NCONV) + bid) % (unsigned)(NR - 1)) + 1;
      __builtin_amdgcn_s_setprio(3);
      const __amdgpu_buffer_rsrc_t rs_d = __builtin_amdgcn_make_buffer_rsrc((void*)p.ws[dk], 0, (int)WS_NEED, RSRC3);
      const int q_raw = 3200 + lane * 16;
      const int q_long = lane < 32 ? q_raw : ((lane - 32) * 16) | (lane < 40 ? 0 : OOR);
      const int q_short = q_long | ((lane & 60) == 32 ? OOR : 0);
      unsigned ends = 0;
      int drop = OOR;
      asm volatile("" : "+s"(drop));
      for (unsigned n = ~0u; ends < (unsigned)NCONV; ++n) {
        unsigned h, pos, spins = 0;
        for (;;) {
          asm volatile("" ::: "memory");
          const unsigned long long r = lds_ld64(smem, L_RDY + (int)n * 8);
          h = __builtin_amdgcn_readfirstlane((unsigned)r);
          pos = __builtin_amdgcn_readfirstlane((unsigned)(r >> 32));
          if ((h | (unsigned)drop) != 0u) break;
          if ((++spins & 255u) == 0u && now() - t0 > 2 * DEADLINE) return;
          __builtin_amdgcn_s_sleep(1);
        }
        asm volatile("" ::: "memory");
        if (drop) pos = 0;
        if (pos >= (unsigned)NPIECE) {
          ends += pos == R_END ? 1u : 0u;
          lds_st(smem, L_PROG + (dk - 1) * 4, n + 1u);
          continue;
        }
        u4 o[4];
        const int q = (h & H_PACKED) ? ((h & H_LONG) ? q_long : q_short) : q_raw;
#pragma unroll
        for (int i = 0; i < 3; ++i) o[i] = *(const u4*)(smem + (n & (RING - 1)) * PS + 128 + i * 1024 + lane * 16);
        o[3] = *(const u4*)(smem + (n & (RING - 1)) * PS + (q & 0xFFFF));
        asm volatile("" ::: "memory");
        lds_st(smem, L_PROG + (dk - 1) * 4, n + 1u);
        const int dof = (int)slot_off(piece_at(pos), (h >> 17) & 1u) | drop;
#pragma unroll
        for (int i = 0; i < 4; ++i) {
          asm volatile("s_waitcnt vmcnt(%0)" :: "n"(SLIDE) : "memory");
          st128(o[i], rs_d, dof + (i < 3 ? 128 + i * 1024 + lane * 16 : q));
          if (i == HINT_POS && !(h & H_PACKED)) __builtin_amdgcn_raw_buffer_store_b32(h, rs_d, (dof + 124) | lane0, 0, AUX_SYS);
        }
        drop = 0;
      }
      return;
    }
    for (unsigned n = (unsigned)wave + NCONV; ; n += NCONV) {
      unsigned spins = 0;
      for (;;) {
        asm volatile("" ::: "memory");
        const unsigned pr = lds_ld(smem, L_PROG + (lane < NR - 1 ? lane : 0) * 4);
        if (__builtin_amdgcn_ballot_w64(pr + FETCH_AHEAD < n) == 0ull) break;
        if ((++spins & 255u) == 0u && now() - t0 > 2 * DEADLINE) return;
        __builtin_amdgcn_s_sleep(2);
      }
      const int raw = __builtin_amdgcn_raw_ptr_buffer_atomic_add_i32(1, rs_my, lane0, (int)TKT_OFF, 0);
      const unsigned pos = ((unsigned)__builtin_amdgcn_readfirstlane(raw) & (NPIECE - 1)) + NCONV * NPW;
      if (pos >= (unsigned)NPIECE) {
        lds_st64(smem, L_RDY + (int)(n + NCONV) * 8, 1ull | ((unsigned long long)R_SKIP << 32));
        lds_st64(smem, L_RDY + (int)n * 8, 1ull | ((unsigned long long)R_END << 32));
        break;
      }
      const int so = (int)slot_off(piece_at(pos), par);
      for (;;) {
        u4 o[4];
        const unsigned hv = __builtin_amdgcn_raw_buffer_load_b32(rs_my, (lane & 31) * 4, so, AUX_SYS);
#pragma unroll
        for (int i = 0; i < 4; ++i)
          o[i] = __builtin_amdgcn_raw_buffer_load_b128(rs_my, lane * 16 + i * 1024, so + 128, AUX_SYS);
        const unsigned h = __builtin_amdgcn_readlane(hv, 31);
        unsigned mn = (h >> 17) == tag ? ((h & H_PACKED) ? hv : h) : 0u;
        if (pos == (unsigned)NPIECE - 1u) {
          const unsigned z = __builtin_amdgcn_raw_buffer_load_b32(rs_my, 0, (int)ZCNT_OFF, AUX_SYS);
          if ((int)(z - call * (unsigned)NGW) < 0) mn = 0u;
        }
#pragma unroll
        for (int i = 0; i < 3; ++i) mn = min4(o[i], mn);
        if (lane < ((h & H_PACKED) ? 32 : 64)) mn = min4(o[3], mn);
        if (__builtin_amdgcn_ballot_w64(mn == 0u) == 0ull) {
          *(unsigned*)(smem + (n & (RING - 1)) * PS + (lane & 31) * 4) = hv;
#pragma unroll
          for (int i = 0; i < 4; ++i) *(u4*)(smem + (n & (RING - 1)) * PS + 128 + i * 1024 + lane * 16) = o[i];
          asm volatile("" ::: "memory");
          lds_st64(smem, L_RDY + (int)n * 8, (unsigned long long)h | ((unsigned long long)pos << 32));
          break;
        }
        if ((++spins & 63u) == 0u && now() - t0 > DEADLINE) return;
        __builtin_amdgcn_s_sleep(4);
      }
    }
    if (wave == 0) { STAMP(8); STAMPV(0, ts00); STAMPV(6, ts0); }
    return;
  }

  const unsigned gi = (bid >> 3) - (unsigned)(NPW / 8);
  const unsigned ti = gi >> 1;
  const unsigned tj = ((bid - ti) & 7u) + (gi & 1u) * 8u;
  const bool hasx = ti < 8u;
  const int wk = wave;
  const int wr = (wk >> 1) & 1, wc = wk & 1;
  float* outp = p.out;
  asm volatile("" : "+s"(outp));
  f4 acc[4][4], acx[2];
  int odrop = 0;

  if (wave >= 4) {
    const unsigned k = (unsigned)(wave - 4);
    const unsigned src = (rank + k) & 7u;
    const unsigned dx = DATA_OFF + par * PAR_BYTES + src * SRC_BYTES + ti * (2u * PS);
    const unsigned de = DATA_OFF + par * PAR_BYTES + src * SRC_BYTES + (30u + (ti >> 2)) * PS;
    const unsigned dw = DATA_OFF + par * PAR_BYTES + TYPE_BYTES + src * SRC_BYTES + tj * (2u * PS);
    const int fo = (int)((lane == 4 && hasx ? de : ((lane & 2) ? dw : dx) + (lane & 1u) * PS) + 124u);
    char* const slot = smem + k * L_SLOT + lane * 16;
    struct Piece { u4 v[4]; unsigned x; };
    const int x_long = lane < 62 ? lane * 2 : OOR, x_short = lane < 32 ? OOR : x_long;
    auto fetch = [&](Piece& c, unsigned h, unsigned so) INL {
#pragma unroll
      for (int i = 0; i < 3; ++i) c.v[i] = __builtin_amdgcn_raw_buffer_load_b128(rs_my, lane * 16 + 128 + i * 1024, (int)so, AUX_SYS);
      c.v[3] = __builtin_amdgcn_raw_buffer_load_b128(rs_my, (lane << ((h & H_PACKED) ? 3 : 4)) + 3200, (int)so, AUX_SYS);
      c.x = __builtin_amdgcn_raw_buffer_load_b16(rs_my, (h & H_PACKED) ? ((h & H_LONG) ? x_long : x_short) : OOR, (int)so, AUX_SYS);
    };
    auto put = [&](const Piece& c, unsigned h, int lo, unsigned mn) INL -> unsigned {
      if (h & H_PACKED) {
        const unsigned bb = ((h >> 10) & 31u) * 0x04000400u;
#pragma unroll
        for (int g = 0; g < 2; ++g) {
          const unsigned T[7] = {c.v[g][0], c.v[g][1], c.v[g][2], c.v[g][3], c.v[2][2 * g], c.v[2][2 * g + 1], c.v[3][g]};
          unsigned d[8], z = ((T[6] >> 1) & 0x10001000u) | ((T[6] << 1) & 0x80008000u);
#pragma unroll
          for (int j = 0; j < 7; ++j) {
            d[j] = (T[j] & 0x9FFF9FFFu) + bb;
            if (j < 6) z |= (T[j] >> (13 - 2 * j)) & (0x00030003u << (2 * j));
            mn = min(mn, T[j]);
          }
          d[7] = z + bb;
          *(u4*)(slot + lo + g * 2048) = u4{d[0], d[1], d[2], d[3]};
          *(u4*)(slot + lo + g * 2048 + 1024) = u4{d[4], d[5], d[6], d[7]};
        }
        asm volatile("" ::: "memory");
        if (c.x & 0x6000u) lds_xor(smem, (int)(k * L_SLOT + lo + (c.x & 0xFFCu)), (c.x & 0x6000u) << ((c.x << 4) & 31u));
        return min(mn, c.x | (unsigned)(((h & H_LONG) ? x_long : x_short) & OOR));
      }
#pragma unroll
      for (int i = 0; i < 4; ++i) { *(u4*)(slot + lo + i * 1024) = c.v[i]; mn = min4(c.v[i], mn); }
      return mn;
    };
    const __amdgpu_buffer_rsrc_t rs_z = __builtin_amdgcn_make_buffer_rsrc(
        (void*)(myws + DATA_OFF + (par ^ 1u) * PAR_BYTES), 0, (int)PAR_BYTES, RSRC3);
    constexpr unsigned ZQ = (PAR_BYTES / 1024 + NGW * 8 - 1) / (NGW * 8);
    unsigned zq = (mode & 1u) ? 0u : ZQ + 1u;
    auto clear = [&](bool all) INL {
      while (zq < ZQ) {
        st128(u4{0u, 0u, 0u, 0u}, rs_z, (int)(((bid - NPW) * 8u + k + zq * (NGW * 8)) * 1024u) + lane * 16);
        ++zq;
        if (!all) return;
      }
      asm volatile("s_waitcnt vmcnt(0)" ::: "memory");
      unsigned n = 0;
      if (lane == 0) n = __hip_atomic_fetch_add((unsigned*)__builtin_assume_aligned(smem + L_MISC + 4, 4), 1u, __ATOMIC_RELAXED, __HIP_MEMORY_SCOPE_WORKGROUP);
      (void)__builtin_amdgcn_raw_ptr_buffer_atomic_add_i32(1, rs_my, n == 7u ? 0 : OOR, (int)ZCNT_OFF, 0);
      zq = ZQ + 1u;
      if (k == 0u) STAMP(3);
    };
    const unsigned long long t0 = now();
    unsigned retries = 0;
    bool dead = false;
    unsigned got = 0, spins = 0, rdy = 0;
    unsigned want = 0;
    unsigned f = 0, fn = 0;
    for (unsigned s = (mode & 2u) ? 0u : 8u; s < 8u; ) {
      {
        asm volatile("" ::: "memory");
        const bool ahead = (rdy & 31u) == 31u;
        const unsigned fnew = __builtin_amdgcn_raw_buffer_load_b32(rs_my, (ahead ? fn | (s == 7u ? 1u : 0u) : f) ? OOR : fo,
                                                                   (int)((s + (ahead ? 1u : 0u)) * SLICE_BYTES), AUX_POLL);
        if (want) {
          Piece a[2], b[2];
          u4 ae = u4{1u, 1u, 1u, 1u};
          const unsigned h0 = __builtin_amdgcn_readlane(f, 0), h1 = __builtin_amdgcn_readlane(f, 1);
          const unsigned h2 = __builtin_amdgcn_readlane(f, 2), h3 = __builtin_amdgcn_readlane(f, 3);
          if (want & 1u) {
            fetch(a[0], h0, dx + s * SLICE_BYTES);
            fetch(a[1], h1, dx + s * SLICE_BYTES + PS);
            if (hasx) ae = __builtin_amdgcn_raw_buffer_load_b128(rs_my, lane * 16 + 128, (int)(de + s * SLICE_BYTES + (ti & 3u) * 1024u), AUX_SYS);
          }
          if (want & 2u) {
            fetch(b[0], h2, dw + s * SLICE_BYTES);
            fetch(b[1], h3, dw + s * SLICE_BYTES + PS);
          }
          if (s > 0) {
            unsigned sp2 = 0;
            for (;;) {
              asm volatile("" ::: "memory");
              const unsigned cv = lds_ld(smem, L_CONS + (int)k * 16 + (lane & 3) * 4);
              if (__builtin_amdgcn_ballot_w64(cv < s) == 0ull) break;
              if ((++sp2 & 255u) == 0u && (lds_ld(smem, L_MISC) != 0u || now() - t0 > DEADLINE)) { dead = true; break; }
              __builtin_amdgcn_s_sleep(1);
            }
            if (dead) break;
          }
          asm volatile("" ::: "memory");
          if (want & 1u) {
            const unsigned e2 = __builtin_amdgcn_readlane(f, 4) & 1023u;
            lds_st(smem, L_LANDED + (int)k * 16 + 8, e2);
            unsigned mn = put(a[0], h0, 0, 0xFFFFFFFFu);
            mn = put(a[1], h1, 4096, mn);
            *(u4*)(slot + 16384) = ae;
            mn = min4(ae, mn);
            asm volatile("" ::: "memory");
            if (__builtin_amdgcn_ballot_w64(mn == 0u) == 0ull) {
              lds_st(smem, L_LANDED + (int)k * 16, landed(s, h0 & 1023u, h1 & 1023u, e2));
              got |= 1u;
            } else ++retries;
          }
          if (want & 2u) {
            unsigned mn = put(b[0], h2, 8192, 0xFFFFFFFFu);
            mn = put(b[1], h3, 12288, mn);
            asm volatile("" ::: "memory");
            if (__builtin_amdgcn_ballot_w64(mn == 0u) == 0ull) {
              lds_st(smem, L_LANDED + (int)k * 16 + 4, landed(s, h2 & 1023u, h3 & 1023u, EBIAS));
              got |= 2u;
            } else ++retries;
          }
          if ((retries & 1023u) == 1023u && now() - t0 > DEADLINE) { dead = true; break; }
        }
        const unsigned fv = (fnew >> 17) == tag ? fnew : 0u;
        if (ahead) fn |= fv; else f |= fv;
        if (got == 3u) {
          ++s; got = 0; spins = 0; f = fn; fn = 0;
          if (s == 8u) break;
          if (s == (unsigned)ZFLUSH && zq <= ZQ) clear(true);
        }
        rdy = (unsigned)__builtin_amdgcn_ballot_w64(f != 0u);
        want = (((rdy & 19u) == 19u ? 1u : 0u) | ((rdy & 12u) == 12u ? 2u : 0u)) & ~got;
        if (want == 0u) {
          if ((++spins & 63u) == 0u) {
            const unsigned tm = __builtin_amdgcn_readfirstlane(__builtin_amdgcn_raw_buffer_load_b32(rs_my, (int)TMO_OFF, 0, AUX_SYS));
            if (tm == call || now() - t0 > DEADLINE) { dead = true; break; }
          }
          if (zq <= ZQ && (s > 0u || spins > (unsigned)ZSTART)) clear(false);
          __builtin_amdgcn_s_sleep(POLL_SLEEP);
        }
      }
    }
    if (zq <= ZQ && !dead) clear(true);
    if (!(mode & 2u)) return;
    if (dead) {
      __builtin_amdgcn_raw_buffer_store_b32(call, rs_my, lane0, (int)TMO_OFF, AUX_SYS);
      lds_st(smem, L_MISC, 1u);
    }
    STAMP(8 + k);
    STAMPV(16 + k, retries);
    STAMPV(24 + k, tseen);
    if (k != 0u) return;
    odrop = OOR;
#pragma unroll
    for (int a = 0; a < 4; ++a)
#pragma unroll
      for (int b = 0; b < 4; ++b) acc[a][b] = f4{0.f, 0.f, 0.f, 0.f};
    acx[0] = acx[1] = f4{0.f, 0.f, 0.f, 0.f};
  } else {
  if (wk == 0) STAMPV(4, ts00);
  if (wk == 0) STAMP(2);
  if (!(mode & 2u)) return;
#pragma unroll
  for (int a = 0; a < 4; ++a)
#pragma unroll
    for (int b = 0; b < 4; ++b) acc[a][b] = f4{0.f, 0.f, 0.f, 0.f};
  acx[0] = acx[1] = f4{0.f, 0.f, 0.f, 0.f};

  unsigned done = 0;
  unsigned nd = 0;
  const char* const abase = smem + wr * 4096 + lane * 16;
  const char* const bbase = smem + 8192 + wc * 4096 + lane * 16;
  const char* const ebase = smem + 16384 + lane * 16;
  const int cons = L_CONS + (lane & 7) * 16 + wk * 4;
  const unsigned long long t0 = now();
  struct Frag { h8 b[4], a[2], e; };
  h8 af[2];
  Frag f0, f1;
  unsigned lx, lw;
  auto peek = [&]() INL {
    asm volatile("" ::: "memory");
    const unsigned long long l = lds_ld64(smem, L_LANDED + (lane & 7) * 16);
    lx = (unsigned)l; lw = (unsigned)(l >> 32);
    asm volatile("" ::: "memory");
  };
  auto take = [&](int k) INL {
    asm volatile("" ::: "memory");
    done += (lane & 7) == k ? 1u : 0u;
    lds_st(smem, cons, done);
    asm volatile("" ::: "memory");
  };
  auto read = [&](Frag& fn, int k) INL {
    asm volatile("" ::: "memory");
#pragma unroll
    for (int i = 0; i < 4; ++i) fn.b[i] = *(const h8*)(bbase + k * L_SLOT + i * 1024);
#pragma unroll
    for (int i = 0; i < 2; ++i) af[i] = *(const h8*)(abase + k * L_SLOT + i * 1024);
#pragma unroll
    for (int i = 0; i < 2; ++i) fn.a[i] = *(const h8*)(abase + k * L_SLOT + 2048 + i * 1024);
    fn.e = *(const h8*)(ebase + k * L_SLOT);
    asm volatile("" ::: "memory");
  };
  enum { W_GO = 0, W_END, W_SLOW, W_DEAD };
  auto wait_and_read = [&](Frag& fn) INL -> int {
    if (nd == 64u) return W_END;
    unsigned rdy, spins = 0;
    for (;;) {
      peek();
      rdy = (unsigned)__builtin_amdgcn_ballot_w64(min(lx & 255u, lw & 255u) > done) & 255u;
      if (rdy) break;
      if (__builtin_amdgcn_ballot_w64(min(lx, lw) >> 28 > done) != 0ull) return W_SLOW;
      if ((++spins & 63u) == 0u && (lds_ld(smem, L_MISC) != 0u || now() - t0 > 2 * DEADLINE)) return W_DEAD;
      __builtin_amdgcn_s_sleep(1);
    }
    const int k = __builtin_ctz(rdy);
    read(fn, k);
    take(k);
    peek();
    ++nd;
    return W_GO;
  };
#define SB() __builtin_amdgcn_sched_barrier(0)
#define MF(A, B) acc[A][B] = __builtin_amdgcn_mfma_f32_16x16x32_f16((A) < 2 ? af[(A) & 1] : f.a[(A) & 1], f.b[B], acc[A][B], 0, 0, 0); SB()
  auto body = [&](auto xs, Frag& f, Frag& fn) INL -> int {
    constexpr int XS = decltype(xs)::value;
    SB();
    MF(0, 0); MF(0, 1);
    const unsigned rdy = (unsigned)__builtin_amdgcn_ballot_w64(min(lx & 255u, lw & 255u) > done) | 256u;
    SB();
    MF(0, 2);
    const int k = __builtin_ctz(rdy);
    const int ko = (k & 7) * L_SLOT;
    const char* const bp = bbase + ko;
    const char* const ap = abase + ko;
    SB();
    MF(0, 3);
    asm volatile("" ::: "memory");
    fn.b[0] = *(const h8*)(bp); SB();
    MF(1, 0);
    fn.b[1] = *(const h8*)(bp + 1024); SB();
    MF(1, 1);
    fn.b[2] = *(const h8*)(bp + 2048); SB();
    MF(1, 2);
    fn.b[3] = *(const h8*)(bp + 3072); SB();
    MF(1, 3);
    af[0] = *(const h8*)(ap); SB();
    MF(2, 0);
    af[1] = *(const h8*)(ap + 1024); SB();
    MF(2, 1);
    fn.a[0] = *(const h8*)(ap + 2048); SB();
    MF(2, 2);
    fn.a[1] = *(const h8*)(ap + 3072); SB();
    MF(2, 3);
    if (XS) fn.e = *(const h8*)(ebase + ko);
    SB();
    MF(3, 0);
    take(k); SB();
    MF(3, 1);
    peek(); SB();
    MF(3, 2); MF(3, 3);
    if (XS) {
      acx[0] = __builtin_amdgcn_mfma_f32_16x16x32_f16(f.e, f.b[XS == 1 ? 0 : 2], acx[0], 0, 0, 0);
      acx[1] = __builtin_amdgcn_mfma_f32_16x16x32_f16(f.e, f.b[XS == 1 ? 1 : 3], acx[1], 0, 0, 0);
      SB();
    }
    if (__builtin_expect(k == 8, 0)) return wait_and_read(fn);
    ++nd;
    return W_GO;
  };
  auto fast = [&](auto xs) INL -> int {
    int st = wait_and_read(f0);
    while (st == W_GO) {
      st = body(xs, f0, f1);
      if (st != W_GO) break;
      st = body(xs, f1, f0);
    }
    return st;
  };
  int st = !hasx ? fast(IC<0>{}) : wr == 0 ? fast(IC<1>{}) : fast(IC<2>{});
  for (; st == W_SLOW && nd < 64u; ++nd) {
    unsigned rdy, spins = 0;
    for (;;) {
      peek();
      rdy = (unsigned)__builtin_amdgcn_ballot_w64(min(lx, lw) >> 28 > done) & 255u;
      if (rdy) break;
      if ((++spins & 63u) == 0u && (lds_ld(smem, L_MISC) != 0u || now() - t0 > 2 * DEADLINE)) { st = W_DEAD; rdy = 1u; break; }
      __builtin_amdgcn_s_sleep(1);
    }
    const int k = __builtin_ctz(rdy);
    const unsigned se = __builtin_amdgcn_readfirstlane(lds_ld(smem, L_LANDED + k * 16 + 8));
    read(f0, k);
    take(k);
    const unsigned sx = __builtin_amdgcn_readlane(lx, k), sw = __builtin_amdgcn_readlane(lw, k);
    const int ew = (int)((sw >> (8 + 10 * wc)) & 1023u) - 2 * (int)EBIAS;
    const int e = (int)((sx >> (8 + 10 * wr)) & 1023u) + ew;
    const int e2 = (int)se + ew;
#pragma unroll
    for (int a = 0; a < 4; ++a)
#pragma unroll
      for (int b = 0; b < 4; ++b) {
        const f4 t = __builtin_amdgcn_mfma_f32_16x16x32_f16(a < 2 ? af[a & 1] : f0.a[a & 1], f0.b[b], f4{0.f, 0.f, 0.f, 0.f}, 0, 0, 0);
#pragma unroll
        for (int r = 0; r < 4; ++r) acc[a][b][r] += __builtin_ldexpf(t[r], e);
        SB();
      }
    const f4 u0 = __builtin_amdgcn_mfma_f32_16x16x32_f16(f0.e, wr ? f0.b[2] : f0.b[0], f4{0.f, 0.f, 0.f, 0.f}, 0, 0, 0);
    const f4 u1 = __builtin_amdgcn_mfma_f32_16x16x32_f16(f0.e, wr ? f0.b[3] : f0.b[1], f4{0.f, 0.f, 0.f, 0.f}, 0, 0, 0);
#pragma unroll
    for (int r = 0; r < 4; ++r) { acx[0][r] += __builtin_ldexpf(u0[r], e2); acx[1][r] += __builtin_ldexpf(u1[r], e2); }
  }
  const bool dead = st == W_DEAD;
  if (wk == 0) STAMP(5);
  if (dead) {
    const float qn = __builtin_nanf("");
#pragma unroll
    for (int a = 0; a < 4; ++a)
#pragma unroll
      for (int b = 0; b < 4; ++b) acc[a][b] = f4{qn, qn, qn, qn};
    acx[0] = acx[1] = f4{qn, qn, qn, qn};
  }
  }

  const __amdgpu_buffer_rsrc_t rs_o = __builtin_amdgcn_make_buffer_rsrc((void*)outp, 0, MM * NN * 4, RSRC3);
  const float K1 = -2.0f * 0.7978845608028654f * 1.4426950408889634f;
  const float K2 = K1 * 0.044715f;
  auto gelu2 = [&](f2 v) -> f2 {
    const f2 t = v * __builtin_elementwise_fma(v * v, f2{K2, K2}, f2{K1, K1});
    const f2 d = f2{__builtin_amdgcn_exp2f(t[0]), __builtin_amdgcn_exp2f(t[1])} + f2{1.f, 1.f};
    return v * f2{__builtin_amdgcn_rcpf(d[0]), __builtin_amdgcn_rcpf(d[1])};
  };
  const int vo = (((lane >> 4) * 4) * (NN * 4) + (lane & 15) * 16) | odrop;
  const int so = (int)((ti * 128u + (unsigned)wr * 64u) * (NN * 4) + (tj * 128u + (unsigned)wc * 64u) * 4u);
#pragma unroll
  for (int a = 0; a < 4; ++a) {
    f2 y[4][2];
#pragma unroll
    for (int b = 0; b < 4; ++b) {
      y[b][0] = gelu2(f2{acc[a][b][0], acc[a][b][1]});
      y[b][1] = gelu2(f2{acc[a][b][2], acc[a][b][3]});
    }
#pragma unroll
    for (int r = 0; r < 4; ++r) {
      const f4 o = f4{y[0][r >> 1][r & 1], y[1][r >> 1][r & 1], y[2][r >> 1][r & 1], y[3][r >> 1][r & 1]};
      st128(__builtin_bit_cast(u4, o), rs_o, vo + so + (a * 16 + r) * (NN * 4));
    }
  }
  if (hasx) {
    const int sx = (int)((15u * 128u + ti * 16u) * (NN * 4) + (tj * 128u + (unsigned)wc * 64u) * 4u) + wr * 8;
    f2 y[2][2];
#pragma unroll
    for (int b = 0; b < 2; ++b) {
      y[b][0] = gelu2(f2{acx[b][0], acx[b][1]});
      y[b][1] = gelu2(f2{acx[b][2], acx[b][3]});
    }
#pragma unroll
    for (int r = 0; r < 4; ++r) {
      const f2 o = f2{y[0][r >> 1][r & 1], y[1][r >> 1][r & 1]};
      __builtin_amdgcn_raw_buffer_store_b64(__builtin_bit_cast(u2, o), rs_o, vo + sx + r * (NN * 4), 0, AUX_SYS);
    }
  }
}

}

extern "C" void kernel_launch(int rank, int nranks, ncclComm_t comm,
                              hipStream_t stream,
                              void* const* d_in, const int* in_sizes, int n_in,
                              void* d_out, int out_size,
                              const int* in_dtypes, int out_dtype,
                              void* const* d_out_all,
                              void* const* d_ws_all, size_t ws_bytes) {
  (void)comm; (void)in_dtypes; (void)out_dtype; (void)d_out_all;
  if (nranks != NR || (unsigned)rank >= (unsigned)NR || n_in < 2 || ws_bytes < (size_t)WS_NEED ||
      in_sizes[0] != MM * KS || in_sizes[1] != KS * NN || out_size != MM * NN) {
    fprintf(stderr, "kernel_launch: unexpected problem shape\n");
    return;
  }
  static int ncu = 0;
  if (ncu == 0) {
    int dev = 0, n = 0;
    HIP_CHECK(hipGetDevice(&dev));
    HIP_CHECK(hipDeviceGetAttribute(&n, hipDeviceAttributeMultiprocessorCount, dev));
    ncu = n > 0 ? n : 1;
  }
  Params p;
  p.x = (const float*)d_in[0];
  p.w = (const float*)d_in[1];
  p.out = (float*)d_out;
  for (int r = 0; r < NR; ++r) p.ws[r] = (char*)d_ws_all[(rank + r) % NR];
  p.my = p.ws[0];
  p.rank = rank;
  if (ncu >= NWG) {
    p.mode = 3;
    hipLaunchKernelGGL(fused_gemm_ag, dim3(NWG), dim3(NWAVE * 64), 0, stream, p);
  } else {
    p.mode = 1;
    hipLaunchKernelGGL(fused_gemm_ag, dim3(NWG), dim3(NWAVE * 64), 0, stream, p);
    p.mode = 2;
    hipLaunchKernelGGL(fused_gemm_ag, dim3(NWG), dim3(NWAVE * 64), 0, stream, p);
  }
  HIP_CHECK(hipGetLastError());
}
```

```cpp
#include <hip/hip_runtime.h>
#include <rccl/rccl.h>
#include <cstdio>
#include <cstdint>

#define HIP_CHECK(expr) do { hipError_t _e = (expr); if (_e != hipSuccess) { \
    fprintf(stderr, "HIP error %s at %s:%d\n", hipGetErrorString(_e), __FILE__, __LINE__); \
    return; } } while (0)

typedef _Float16 h8 __attribute__((ext_vector_type(8)));
typedef _Float16 h2 __attribute__((ext_vector_type(2)));
typedef float f4 __attribute__((ext_vector_type(4)));
typedef float f2 __attribute__((ext_vector_type(2)));
typedef unsigned u4 __attribute__((ext_vector_type(4)));
typedef unsigned u2 __attribute__((ext_vector_type(2)));
typedef unsigned short us2 __attribute__((ext_vector_type(2)));

#define SLIDE 4
#define CONV_SHIFT 4
#define CONV_SLEEP 4
#define CONV_STAG 1
#define EARLY 1
#define HINT_POS 2
#define CONV_BASE 48
#define FETCH_AHEAD 3
#define AUX_POLL 17
#define POLL_SLEEP 2
#define ZFLUSH 5
#define ZSTART 4
#define RAW_HEAD 32
#define RAW_TAIL 48
#define WSLEEP 1
#define LPRIO 2
#define LEAD 4
#define LEAD_FROM 0
#define EARLY_TAIL 0
#define HINT_EARLY 0

namespace {

constexpr int NR = 8;
constexpr int MM = 2048, NN = 2048, KS = 256;
constexpr int NWG = 256;
constexpr int NPW = 16;
constexpr int NGW = NWG - NPW;
constexpr int NWAVE = 12;
constexpr int NCONV = 5;
constexpr int NPIECE = 512;
constexpr int PPW = NPIECE / NPW;

constexpr unsigned CTR_OFF  = 0;
constexpr unsigned TMO_OFF  = 32768;
constexpr unsigned TKT_OFF  = 36992;
constexpr unsigned ZCNT_OFF = 36864;
constexpr unsigned DBG_OFF  = 262144;
constexpr unsigned DATA_OFF = 1u << 20;
constexpr unsigned PS = 4224;
constexpr unsigned SLICE_BYTES = 32 * PS;
constexpr unsigned SRC_BYTES = 8 * SLICE_BYTES;
constexpr unsigned TYPE_BYTES = 8 * SRC_BYTES;
constexpr unsigned PAR_BYTES = 2 * TYPE_BYTES;
constexpr unsigned WS_NEED = DATA_OFF + 2 * PAR_BYTES;

constexpr int AUX_SYS = 17;
constexpr unsigned RSRC3 = 0x00020000u;
constexpr unsigned long long DEADLINE = 100000000ull;
constexpr unsigned CALL_MASK = 0x7FFFu;
constexpr unsigned H_PACKED = 1u << 15, H_LONG = 1u << 16;
constexpr unsigned EBIAS = 512;
constexpr int OOR = 0x40000000;

constexpr int L_SLOT = 17408;
constexpr int L_CTL = NR * L_SLOT;
constexpr int L_LANDED = L_CTL;
constexpr int L_CONS = L_CTL + 128;
constexpr int L_MISC = L_CTL + 256;
constexpr int L_HDR = L_CTL + 512;
constexpr int L_BYTES = L_CTL + 768;
constexpr int RING = 16;
constexpr int L_RDY = RING * (int)PS;
constexpr int L_PROG = L_CTL;
constexpr unsigned R_END = 0xFFFFu, R_SKIP = 0xFFFEu;

struct Params {
  const float* x;
  const float* w;
  char* my;
  int rank;
  int mode;
  char* ws[NR];
  float* out;
};

__device__ __forceinline__ unsigned long long now() { return __builtin_amdgcn_s_memrealtime(); }

__device__ __forceinline__ unsigned pack2(float a, float b) {
  h2 h = {(_Float16)a, (_Float16)b};
  return __builtin_bit_cast(unsigned, h);
}
__device__ __forceinline__ float max3abs(float a, float b, float m) {
  float r;
  asm("v_max3_f32 %0, |%1|, |%2|, %3" : "=v"(r) : "v"(a), "v"(b), "v"(m));
  return r;
}

__device__ __forceinline__ unsigned piece_at(unsigned p) {
  const unsigned s = p >> 6, q = (p >> 1) & 31u, half = p & 1u;
  const unsigned ld = s >= (unsigned)LEAD_FROM ? LEAD : 0u;
  const unsigned type = q < ld ? 1u : q >= 32u - ld ? 0u : (q - ld) & 1u;
  const unsigned b = q < ld ? q : q >= 32u - ld ? q - 17u : (((q - ld) >> 1) + (type ? ld : 15u)) & 15u;
  return (type << 8) | (s << 5) | (b * 2u + half);
}

__device__ __forceinline__ void st128(u4 v, __amdgpu_buffer_rsrc_t rs, int voff) {
  __builtin_amdgcn_raw_buffer_store_b128(v, rs, voff, 0, AUX_SYS);
}

__device__ __forceinline__ unsigned lds_ld(const char* smem, int off) {
  return __hip_atomic_load((const unsigned*)(smem + off), __ATOMIC_RELAXED, __HIP_MEMORY_SCOPE_WORKGROUP);
}
__device__ __forceinline__ unsigned long long lds_ld64(const char* smem, int off) {
  return __hip_atomic_load((const unsigned long long*)(smem + off), __ATOMIC_RELAXED, __HIP_MEMORY_SCOPE_WORKGROUP);
}
__device__ __forceinline__ void lds_st(char* smem, int off, unsigned v) {
  __hip_atomic_store((unsigned*)(smem + off), v, __ATOMIC_RELAXED, __HIP_MEMORY_SCOPE_WORKGROUP);
}
__device__ __forceinline__ void lds_st64(char* smem, int off, unsigned long long v) {
  __hip_atomic_store((unsigned long long*)(smem + off), v, __ATOMIC_RELAXED, __HIP_MEMORY_SCOPE_WORKGROUP);
}
__device__ __forceinline__ void lds_xor(char* smem, int off, unsigned v) {
  (void)__hip_atomic_fetch_xor((unsigned*)(smem + off), v, __ATOMIC_RELAXED, __HIP_MEMORY_SCOPE_WORKGROUP);
}
__device__ __forceinline__ unsigned landed(unsigned s, unsigned h0, unsigned h1, unsigned e2) {
  const unsigned e = ((h0 & 1023u) << 8) | ((h1 & 1023u) << 18);
  unsigned r = ((s + 1u) << 28) | e | (((e ^ (EBIAS * 0x40100u)) | (e2 ^ EBIAS)) == 0u ? s + 1u : 0u);
  asm volatile("" : "+s"(r));
  return r;
}
#define INL __attribute__((always_inline))
template <int V> struct IC { static constexpr int value = V; };
__device__ __forceinline__ unsigned min4(u4 v, unsigned m) {
  asm("v_min3_u32 %0, %1, %2, %0\n\tv_min3_u32 %0, %3, %4, %0" : "+v"(m) : "v"(v[0]), "v"(v[1]), "v"(v[2]), "v"(v[3]));
  return m;
}

#define STAMP(idx) do {} while (0)
#define STAMPV(idx, v) do {} while (0)

__global__ void __launch_bounds__(NWAVE * 64, 1) fused_gemm_ag(Params p) {
  __shared__ __attribute__((aligned(16))) char smem[L_BYTES];
  const int lane = threadIdx.x & 63;
  const int wave = __builtin_amdgcn_readfirstlane(threadIdx.x >> 6);
  const unsigned bid = blockIdx.x;
  const float* px = p.x;
  const float* pw = p.w;
  char* myws = p.my;
  unsigned rank = (unsigned)p.rank, mode = (unsigned)p.mode;
  const bool sendwg = bid < (unsigned)NPW;

  const unsigned cpos = sendwg ? (unsigned)wave * NPW + bid : NCONV * NPW + (unsigned)wave * NGW + (bid - NPW);
  f4 R[4][2];
  auto conv_issue = [&](unsigned cpc) {
    const int q = lane >> 4, c = lane & 15;
    const bool isw = (cpc >> 8) != 0u;
    const unsigned s = (cpc >> 5) & 7u, g = cpc & 31u;
    const int sb = (int)(s << (isw ? 18 : 7)) + (int)(g << (isw ? 8 : 16));
    const int sa = isw ? NN * 4 : 16 * KS * 4;
    const int sbb = isw ? 16 * NN * 4 : 64;
    const int vo = (q << (isw ? 15 : 4)) + (c << (isw ? 4 : 10));
    asm volatile("" : "+s"(px), "+s"(pw), "+s"(myws), "+s"(rank), "+s"(mode));
    const __amdgpu_buffer_rsrc_t rs = __builtin_amdgcn_make_buffer_rsrc((void*)(isw ? pw : px), 0, MM * KS * 4, RSRC3);
#pragma unroll
    for (int a = 0; a < 4; ++a)
#pragma unroll
      for (int b = 0; b < 2; ++b)
        R[a][b] = __builtin_bit_cast(f4, __builtin_amdgcn_raw_buffer_load_b128(rs, vo, sb + a * sa + b * sbb, 0));
  };
  const bool early = sendwg && wave < EARLY;
  if (early) conv_issue(piece_at(cpos & 63u));
  rank &= 7u;
  const unsigned cpc = piece_at(cpos & (NPIECE - 1));
  if (sendwg && !(mode & 1u)) return;
  const __amdgpu_buffer_rsrc_t rs_my = __builtin_amdgcn_make_buffer_rsrc((void*)myws, 0, (int)WS_NEED, RSRC3);
  const int ctr_off = (int)(CTR_OFF + (bid * 16u + (unsigned)wave) * 4u);
  const int lane0 = lane == 0 ? 0 : OOR;
  const bool conv = (mode & 1u) && (sendwg ? wave < NCONV : cpos < (unsigned)NPIECE);
  const bool sender = sendwg && wave >= NCONV;
  unsigned cv = __builtin_amdgcn_raw_buffer_load_b32(rs_my, sender ? OOR : ctr_off, 0, AUX_SYS);
  if (wave == (sendwg ? 5 : 2)) {
    lds_st(smem, L_CTL + lane * 4, 0u);
    lds_st(smem, L_CTL + 256 + lane * 4, 0u);
  }
  if (sendwg && wave >= 6 && wave < 11) *(u4*)(smem + L_RDY + (wave - 6) * 1024 + lane * 16) = u4{0u, 0u, 0u, 0u};
  asm volatile("s_waitcnt lgkmcnt(0)\n\ts_barrier" ::: "memory");
  if (conv && !early) {
    const unsigned nsl = sendwg ? (unsigned)(wave - EARLY) * CONV_STAG : (cpos - CONV_BASE) >> CONV_SHIFT;
    for (unsigned i = 0; i < nsl; ++i) __builtin_amdgcn_s_sleep(CONV_SLEEP);
    conv_issue(cpc);
  }
  asm volatile("" : "+v"(cv));
  const unsigned call = __builtin_amdgcn_readfirstlane(cv) + 1u;
  const unsigned par = call & 1u;
  const unsigned tag = call & CALL_MASK;
  if (!sender && (mode & (!sendwg && wave >= 4 ? 2u : 1u))) __builtin_amdgcn_raw_buffer_store_b32(call, rs_my, lane0, ctr_off, AUX_SYS);
  auto slot_off = [&](unsigned pc, unsigned pr) -> unsigned {
    return DATA_OFF + pr * PAR_BYTES + rank * SRC_BYTES + (pc >> 8) * TYPE_BYTES + (pc & 255u) * PS;
  };
  if (conv) {
    float m = 0.f;
#pragma unroll
    for (int a = 0; a < 4; ++a)
#pragma unroll
      for (int b = 0; b < 2; ++b) {
        m = max3abs(R[a][b][0], R[a][b][1], m);
        m = max3abs(R[a][b][2], R[a][b][3], m);
      }
    int e = 0;
    const bool big = __builtin_amdgcn_ballot_w64(m >= 32768.f) != 0ull;
    const bool small = __builtin_amdgcn_ballot_w64(m >= 0.00390625f) == 0ull && __builtin_amdgcn_ballot_w64(m > 0.f) != 0ull;
    if (big || small) {
      float mf = 0.f;
#pragma unroll
      for (int a = 0; a < 4; ++a)
#pragma unroll
        for (int b = 0; b < 2; ++b)
#pragma unroll
          for (int t = 0; t < 4; ++t) {
            const float av = __builtin_fabsf(R[a][b][t]);
            mf = fmaxf(mf, av < __builtin_inff() ? av : 0.f);
          }
#pragma unroll
      for (int o = 32; o >= 1; o >>= 1) mf = fmaxf(mf, __shfl_xor(mf, o, 64));
      int ee = 0;
      if (mf > 0.f) { (void)__builtin_frexpf(mf, &ee); ee -= 10; }
      e = __builtin_amdgcn_readfirstlane(ee);
#pragma unroll
      for (int a = 0; a < 4; ++a)
#pragma unroll
        for (int b = 0; b < 2; ++b)
#pragma unroll
          for (int t = 0; t < 4; ++t) R[a][b][t] = __builtin_ldexpf(R[a][b][t], -e);
    }
    u4 o[4];
    if ((cpc >> 8) == 0u) {
#pragma unroll
      for (int a = 0; a < 4; ++a)
        o[a] = u4{pack2(R[a][0][0], R[a][0][1]), pack2(R[a][0][2], R[a][0][3]),
                  pack2(R[a][1][0], R[a][1][1]), pack2(R[a][1][2], R[a][1][3])};
    } else {
#pragma unroll
      for (int i = 0; i < 4; ++i)
        o[i] = u4{pack2(R[0][0][i], R[1][0][i]), pack2(R[2][0][i], R[3][0][i]),
                  pack2(R[0][1][i], R[1][1][i]), pack2(R[2][1][i], R[3][1][i])};
    }
    unsigned mn = 0xFFFFFFFFu;
#pragma unroll
    for (int i = 0; i < 4; ++i) mn = min4(o[i], mn);
    if (__builtin_amdgcn_ballot_w64(mn == 0u) != 0ull) {
#pragma unroll
      for (int i = 0; i < 4; ++i)
#pragma unroll
        for (int t = 0; t < 4; ++t) o[i][t] = o[i][t] == 0u ? 0x8000u : o[i][t];
    }
    unsigned h = (tag << 17) | (unsigned)(e + (int)EBIAS);
    char* const hdr = smem + (sendwg ? wave * (int)PS : L_HDR + wave * 128);
    if (cpos >= (unsigned)RAW_HEAD && cpos < (unsigned)(NPIECE - RAW_TAIL) && (cpc & 0x11Eu) != 0x1Eu) {
      *(unsigned*)(hdr + (lane & 31) * 4) = 0x80008000u;
      const unsigned eh = (pack2(__builtin_ldexpf(m, -e), 0.f) >> 10) & 31u;
      unsigned em = 0;
#pragma unroll
      for (unsigned bit = 16; bit; bit >>= 1) em |= __builtin_amdgcn_ballot_w64(eh >= (em | bit)) != 0ull ? bit : 0u;
      const float top = __builtin_bit_cast(float, (em + 112u) << 23), low = __builtin_bit_cast(float, (em + 105u) << 23);
      int nt = 0, nl = 0;
#pragma unroll
      for (int a = 0; a < 4; ++a)
#pragma unroll
        for (int t = 0; t < 4; ++t) {
          nt += __builtin_popcountll(__builtin_amdgcn_ballot_w64(__builtin_fabsf(R[a][0][t]) >= top));
          nl += __builtin_popcountll(__builtin_amdgcn_ballot_w64(__builtin_fabsf(R[a][0][t]) < low));
        }
      const unsigned base = (unsigned)max((int)em - (2 * nt < nl ? 8 : 7), 0);
      const unsigned bb = base * 0x04040404u;
      unsigned T[2][8], M[2] = {0u, 0u};
      mn = 0xFFFFFFFFu;
#pragma unroll
      for (int g = 0; g < 2; ++g) {
#pragma unroll
        for (int j = 0; j < 4; ++j) {
          const unsigned d0 = o[2 * g + (j >> 1)][(2 * j) & 3], d1 = o[2 * g + (j >> 1)][(2 * j + 1) & 3];
          const unsigned hi = __builtin_amdgcn_perm(d1, d0, 0x07050301u);
          const unsigned v = ((hi | 0x80808080u) - bb) & 0x1F1F1F1Fu;
          const unsigned xm = ((v + bb) ^ hi) & 0x60606060u;
          M[g] |= j < 3 ? xm >> (5 - 2 * j) : xm << 1;
          T[g][j] = __builtin_amdgcn_perm(d1, d0, 0x06040200u);
          T[g][4 + j] = v | (hi & 0x80808080u);
        }
        T[g][4] |= (T[g][7] << 5) & 0x60606060u;
        T[g][5] |= (T[g][7] << 3) & 0x60606060u;
        T[g][6] |= ((T[g][7] << 1) & 0x20202020u) | ((T[g][7] >> 1) & 0x40404040u);
#pragma unroll
        for (int j = 0; j < 7; ++j) mn = min(mn, T[g][j]);
      }
      unsigned cnt = __builtin_amdgcn_ballot_w64(mn == 0u) != 0ull ? 64u : 0u;
      asm volatile("" ::: "memory");
#pragma unroll
      for (int g = 0; g < 2; ++g) {
        unsigned mg = M[g];
        while (cnt <= 62u) {
          const unsigned long long bal = __builtin_amdgcn_ballot_w64(mg != 0u);
          if (bal == 0ull) break;
          const unsigned n = cnt + __builtin_amdgcn_mbcnt_hi((unsigned)(bal >> 32), __builtin_amdgcn_mbcnt_lo((unsigned)bal, 0u));
          const unsigned b = (unsigned)__builtin_ctz(mg | 0x80000000u) & 30u;
          const unsigned en = 0x8000u | (((mg >> b) & 3u) << 13) | (unsigned)(g << 11) | ((b & 4u) << 8) | (unsigned)(lane << 4) | ((b & 2u) << 2) | ((b & 16u) >> 2) | ((b >> 3) & 1u);
          if (mg != 0u && n < 62u) *(unsigned short*)(hdr + (61u - n) * 2u) = (unsigned short)en;
          mg &= ~(3u << b);
          cnt += (unsigned)__builtin_popcountll(bal);
        }
      }
      if (cnt <= 62u) {
        h |= (base << 10) | H_PACKED | (cnt > 30u ? H_LONG : 0u);
        o[0] = u4{T[0][0], T[0][1], T[0][2], T[0][3]};
        o[1] = u4{T[1][0], T[1][1], T[1][2], T[1][3]};
        o[2] = u4{T[0][4], T[0][5], T[1][4], T[1][5]};
        o[3][0] = T[0][6];
        o[3][1] = T[1][6];
      }
    }
    const bool packed = (h & H_PACKED) != 0u;
    asm volatile("" ::: "memory");
    lds_st(hdr, 124, h);
    if (sendwg) {
#pragma unroll
      for (int i = 0; i < 3; ++i) *(u4*)(hdr + 128 + i * 1024 + lane * 16) = o[i];
      if (packed) *(u2*)(hdr + 3200 + lane * 8) = u2{o[3][0], o[3][1]};
      else *(u4*)(hdr + 3200 + lane * 16) = o[3];
      asm volatile("" ::: "memory");
      lds_st64(smem, L_RDY + wave * 8, (unsigned long long)h | ((unsigned long long)cpos << 32));
      if (wave == 0) STAMP(5);
      asm volatile("" ::: "memory");
    }
    const int dof = (int)slot_off(cpc, par);
    const unsigned hv = lds_ld(hdr, (lane & 31) * 4);
#pragma unroll
    for (int i = 0; i < 3; ++i) st128(o[i], rs_my, dof + 128 + i * 1024 + lane * 16);
    if (packed) __builtin_amdgcn_raw_buffer_store_b64(u2{o[3][0], o[3][1]}, rs_my, dof + 3200 + lane * 8, 0, AUX_SYS);
    else st128(o[3], rs_my, dof + 3200 + lane * 16);
    __builtin_amdgcn_raw_buffer_store_b32(hv, rs_my, (packed ? lane < 32 : lane == 31) ? dof + lane * 4 : OOR, 0, AUX_SYS);
  }

  if (sendwg) {
    const unsigned long long t0 = now();
    if (sender) {
      const int dk = (int)(((unsigned)(wave - NCONV) + bid) % (unsigned)(NR - 1)) + 1;
      __builtin_amdgcn_s_setprio(3);
      const __amdgpu_buffer_rsrc_t rs_d = __builtin_amdgcn_make_buffer_rsrc((void*)p.ws[dk], 0, (int)WS_NEED, RSRC3);
      const int q_raw = 3200 + lane * 16;
      const int q_long = lane < 32 ? q_raw : ((lane - 32) * 16) | (lane < 40 ? 0 : OOR);
      const int q_short = q_long | ((lane & 60) == 32 ? OOR : 0);
      unsigned ends = 0;
      int drop = OOR;
      asm volatile("" : "+s"(drop));
      for (unsigned n = ~0u; ends < (unsigned)NCONV; ++n) {
        unsigned h, pos, spins = 0;
        for (;;) {
          asm volatile("" ::: "memory");
          const unsigned long long r = lds_ld64(smem, L_RDY + (int)n * 8);
          h = __builtin_amdgcn_readfirstlane((unsigned)r);
          pos = __builtin_amdgcn_readfirstlane((unsigned)(r >> 32));
          if ((h | (unsigned)drop) != 0u) break;
          if ((++spins & 255u) == 0u && now() - t0 > 2 * DEADLINE) return;
          __builtin_amdgcn_s_sleep(1);
        }
        asm volatile("" ::: "memory");
        if (drop) pos = 0;
        if (pos >= (unsigned)NPIECE) {
          ends += pos == R_END ? 1u : 0u;
          lds_st(smem, L_PROG + (dk - 1) * 4, n + 1u);
          continue;
        }
        u4 o[4];
        const int q = (h & H_PACKED) ? ((h & H_LONG) ? q_long : q_short) : q_raw;
#pragma unroll
        for (int i = 0; i < 3; ++i) o[i] = *(const u4*)(smem + (n & (RING - 1)) * PS + 128 + i * 1024 + lane * 16);
        o[3] = *(const u4*)(smem + (n & (RING - 1)) * PS + (q & 0xFFFF));
        asm volatile("" ::: "memory");
        lds_st(smem, L_PROG + (dk - 1) * 4, n + 1u);
        const int dof = (int)slot_off(piece_at(pos), (h >> 17) & 1u) | drop;
        const int hp = (h & H_PACKED) ? 9 : pos >= (unsigned)(NPIECE - EARLY_TAIL) ? HINT_EARLY : HINT_POS;
        if (hp == -1) __builtin_amdgcn_raw_buffer_store_b32(h, rs_d, (dof + 124) | lane0, 0, AUX_SYS);
#pragma unroll
        for (int i = 0; i < 4; ++i) {
          asm volatile("s_waitcnt vmcnt(%0)" :: "n"(SLIDE) : "memory");
          st128(o[i], rs_d, dof + (i < 3 ? 128 + i * 1024 + lane * 16 : q));
          if (i == hp) __builtin_amdgcn_raw_buffer_store_b32(h, rs_d, (dof + 124) | lane0, 0, AUX_SYS);
        }
        drop = 0;
      }
      return;
    }
    for (unsigned n = (unsigned)wave + NCONV; ; n += NCONV) {
      unsigned spins = 0;
      for (;;) {
        asm volatile("" ::: "memory");
        const unsigned pr = lds_ld(smem, L_PROG + (lane < NR - 1 ? lane : 0) * 4);
        if (__builtin_amdgcn_ballot_w64(pr + FETCH_AHEAD < n) == 0ull) break;
        if ((++spins & 255u) == 0u && now() - t0 > 2 * DEADLINE) return;
        __builtin_amdgcn_s_sleep(2);
      }
      const int raw = __builtin_amdgcn_raw_ptr_buffer_atomic_add_i32(1, rs_my, lane0, (int)TKT_OFF, 0);
      const unsigned pos = ((unsigned)__builtin_amdgcn_readfirstlane(raw) & (NPIECE - 1)) + NCONV * NPW;
      if (pos >= (unsigned)NPIECE) {
        lds_st64(smem, L_RDY + (int)(n + NCONV) * 8, 1ull | ((unsigned long long)R_SKIP << 32));
        lds_st64(smem, L_RDY + (int)n * 8, 1ull | ((unsigned long long)R_END << 32));
        break;
      }
      const int so = (int)slot_off(piece_at(pos), par);
      for (;;) {
        u4 o[4];
        const unsigned hv = __builtin_amdgcn_raw_buffer_load_b32(rs_my, (lane & 31) * 4, so, AUX_SYS);
#pragma unroll
        for (int i = 0; i < 4; ++i)
          o[i] = __builtin_amdgcn_raw_buffer_load_b128(rs_my, lane * 16 + i * 1024, so + 128, AUX_SYS);
        const unsigned h = __builtin_amdgcn_readlane(hv, 31);
        unsigned mn = (h >> 17) == tag ? ((h & H_PACKED) ? hv : h) : 0u;
        if (pos == (unsigned)NPIECE - 1u) {
          const unsigned z = __builtin_amdgcn_raw_buffer_load_b32(rs_my, 0, (int)ZCNT_OFF, AUX_SYS);
          if ((int)(z - call * (unsigned)NGW) < 0) mn = 0u;
        }
#pragma unroll
        for (int i = 0; i < 3; ++i) mn = min4(o[i], mn);
        if (lane < ((h & H_PACKED) ? 32 : 64)) mn = min4(o[3], mn);
        if (__builtin_amdgcn_ballot_w64(mn == 0u) == 0ull) {
          *(unsigned*)(smem + (n & (RING - 1)) * PS + (lane & 31) * 4) = hv;
#pragma unroll
          for (int i = 0; i < 4; ++i) *(u4*)(smem + (n & (RING - 1)) * PS + 128 + i * 1024 + lane * 16) = o[i];
          asm volatile("" ::: "memory");
          lds_st64(smem, L_RDY + (int)n * 8, (unsigned long long)h | ((unsigned long long)pos << 32));
          break;
        }
        if ((++spins & 63u) == 0u && now() - t0 > DEADLINE) return;
        __builtin_amdgcn_s_sleep(4);
      }
    }
    if (wave == 0) { STAMP(8); STAMPV(0, ts00); STAMPV(6, ts0); }
    return;
  }

  const unsigned gi = (bid >> 3) - (unsigned)(NPW / 8);
  const unsigned ti = gi >> 1;
  const unsigned tj = ((bid - ti) & 7u) + (gi & 1u) * 8u;
  const bool hasx = ti < 8u;
  const int wk = wave;
  const int wr = (wk >> 1) & 1, wc = wk & 1;
  float* outp = p.out;
  asm volatile("" : "+s"(outp));
  f4 acc[4][4], acx[2];
  int odrop = 0;

  if (wave >= 4) {
    const unsigned k = (unsigned)(wave - 4);
    const unsigned src = (rank + k) & 7u;
    const unsigned dx = DATA_OFF + par * PAR_BYTES + src * SRC_BYTES + ti * (2u * PS);
    const unsigned de = DATA_OFF + par * PAR_BYTES + src * SRC_BYTES + (30u + (ti >> 2)) * PS;
    const unsigned dw = DATA_OFF + par * PAR_BYTES + TYPE_BYTES + src * SRC_BYTES + tj * (2u * PS);
    const int fo = (int)((lane == 4 && hasx ? de : ((lane & 2) ? dw : dx) + (lane & 1u) * PS) + 124u);
    char* const slot = smem + k * L_SLOT + lane * 16;
    const unsigned lds_base = (unsigned)(size_t)(__attribute__((address_space(3))) char*)smem;
    struct Piece { u4 v[4]; unsigned x; };
    const int x_long = lane < 62 ? lane * 2 : OOR, x_short = lane < 32 ? OOR : x_long;
    auto fetch = [&](Piece* c, unsigned h0, unsigned h1, unsigned so) INL {
#pragma unroll
      for (int i = 0; i < 6; ++i) c[i & 1].v[i >> 1] = __builtin_amdgcn_raw_buffer_load_b128(rs_my, lane * 16 + 128 + (i >> 1) * 1024, (int)(so + (i & 1) * PS), AUX_SYS);
      asm volatile("" ::: "memory");
#pragma unroll
      for (int i = 0; i < 2; ++i) {
        c[i].x = __builtin_amdgcn_raw_buffer_load_b16(rs_my, x_long, (int)(so + i * PS), AUX_SYS);
        c[i].v[3] = __builtin_amdgcn_raw_buffer_load_b128(rs_my, ((lane * 16) >> (((i ? h1 : h0) >> 15) & 1u)) + 3200, (int)(so + i * PS), AUX_SYS);
      }
    };
    unsigned m60 = 0x60606060u;
    asm volatile("" : "+s"(m60));
    auto put = [&](const Piece& c, unsigned h, int lo, unsigned mn) INL -> unsigned {
      if (__builtin_expect((h & H_PACKED) != 0u, 1)) {
        const unsigned bb = ((h >> 10) & 31u) * 0x04040404u;
#pragma unroll
        for (int g = 0; g < 2; ++g) {
          const unsigned T[7] = {c.v[g][0], c.v[g][1], c.v[g][2], c.v[g][3], c.v[2][2 * g], c.v[2][2 * g + 1], c.v[3][g]};
          unsigned d[8], hi[4], B[3];
#pragma unroll
          for (int j = 0; j < 3; ++j) {
            B[j] = T[4 + j] & m60;
            asm("v_xad_u32 %0, %1, %2, %3" : "=v"(hi[j]) : "v"(T[4 + j]), "v"(B[j]), "s"(bb));
          }
          unsigned u = B[2] >> 1;
          asm("v_lshl_add_u32 %0, %1, 2, %1" : "=v"(u) : "v"(u));
          asm("v_and_or_b32 %0, %1, %2, %3" : "=v"(u) : "v"(u), "s"(0x90909090u), "v"(B[0] >> 5));
          asm("v_add3_u32 %0, %1, %2, %3" : "=v"(hi[3]) : "v"(u), "v"(B[1] >> 3), "s"(bb));
#pragma unroll
          for (int j = 0; j < 4; ++j) {
            d[2 * j] = __builtin_amdgcn_perm(hi[j], T[j], 0x05010400u);
            d[2 * j + 1] = __builtin_amdgcn_perm(hi[j], T[j], 0x07030602u);
          }
#pragma unroll
          for (int j = 0; j < 7; ++j) mn = min(mn, T[j]);
          *(u4*)(slot + lo + g * 2048) = u4{d[0], d[1], d[2], d[3]};
          *(u4*)(slot + lo + g * 2048 + 1024) = u4{d[4], d[5], d[6], d[7]};
        }
        asm volatile("" ::: "memory");
        if (c.x & 0x6000u) lds_xor(smem, (int)(k * L_SLOT + lo + (c.x & 0xFFCu)), (c.x & 0x6000u) << ((c.x << 4) & 31u));
        return min(mn, c.x | (unsigned)(((h & H_LONG) ? x_long : x_short) & OOR));
      }
#pragma unroll
      for (int i = 0; i < 4; ++i) { *(u4*)(slot + lo + i * 1024) = c.v[i]; mn = min4(c.v[i], mn); }
      return mn;
    };
    const __amdgpu_buffer_rsrc_t rs_z = __builtin_amdgcn_make_buffer_rsrc(
        (void*)(myws + DATA_OFF + (par ^ 1u) * PAR_BYTES), 0, (int)PAR_BYTES, RSRC3);
    constexpr unsigned ZQ = (PAR_BYTES / 1024 + NGW * 8 - 1) / (NGW * 8);
    unsigned zq = (mode & 1u) ? 0u : ZQ + 1u;
    auto clear = [&](bool all) INL {
      while (zq < ZQ) {
        st128(u4{0u, 0u, 0u, 0u}, rs_z, (int)(((bid - NPW) * 8u + k + zq * (NGW * 8)) * 1024u) + lane * 16);
        ++zq;
        if (!all) return;
      }
      if (all) asm volatile("s_waitcnt vmcnt(0)" ::: "memory");
      unsigned n = 0;
      if (lane == 0) n = __hip_atomic_fetch_add((unsigned*)__builtin_assume_aligned(smem + L_MISC + 4, 4), 1u, __ATOMIC_RELAXED, __HIP_MEMORY_SCOPE_WORKGROUP);
      (void)__builtin_amdgcn_raw_ptr_buffer_atomic_add_i32(1, rs_my, n == 7u ? 0 : OOR, (int)ZCNT_OFF, 0);
      zq = ZQ + 1u;
      if (k == 0u) STAMP(3);
    };
    const unsigned long long t0 = now();
    unsigned retries = 0;
    enum : unsigned { S_X = 19u, S_W = 12u, S_ANN = 32u, S_FLY = 64u, S_END = 96u };
    bool dead = false;
    unsigned spins = 0, sx = (mode & 2u) ? 0u : 8u, sw = sx, stx = S_X, stw = S_W;
    unsigned h0 = 0, h1 = 0, h2 = 0, h3 = 0, e2 = 0;
    unsigned f = 0, fnew = 0;
    int po = lane < 5 ? fo : OOR;
    const unsigned wl = (lane & 2) ? ~0u : 0u;
    Piece a[2], b[2];
    u4 ae = u4{1u, 1u, 1u, 1u};
    auto poll = [&]() INL {
      fnew = __builtin_amdgcn_raw_buffer_load_b32(rs_my, f ? OOR : po, 0, AUX_POLL);
      asm volatile("" ::: "memory");
    };
    unsigned cv;
    auto top = [&]() INL {
      asm volatile("" ::: "memory");
      __builtin_amdgcn_s_waitcnt(0x0F70);
      cv = lds_ld(smem, L_CONS + (int)k * 16 + (lane & 3) * 4);
      f |= (fnew >> 17) == tag ? fnew : 0u;
      const unsigned rdy = (unsigned)__builtin_amdgcn_ballot_w64(f != 0u);
      if ((rdy & S_X) == stx) {
        h0 = __builtin_amdgcn_readlane(f, 0); h1 = __builtin_amdgcn_readlane(f, 1); e2 = __builtin_amdgcn_readlane(f, 4) & 1023u;
        f &= wl; po += (int)(SLICE_BYTES & ~wl);
        stx = S_ANN;
      }
      if ((rdy & S_W) == stw) {
        h2 = __builtin_amdgcn_readlane(f, 2); h3 = __builtin_amdgcn_readlane(f, 3);
        f &= ~wl; po += (int)(SLICE_BYTES & wl);
        stw = S_ANN;
      }
    };
    auto room = [&](unsigned s) INL {
      if (__builtin_expect(__builtin_amdgcn_ballot_w64(cv < s) != 0ull, 0)) {
        unsigned sp = 0;
        do {
          asm volatile("" ::: "memory");
          cv = lds_ld(smem, L_CONS + (int)k * 16 + (lane & 3) * 4);
          if ((++sp & 1023u) == 0u && (lds_ld(smem, L_MISC) != 0u || now() - t0 > DEADLINE)) { spins = 63u; cv = 8u; }
        } while (__builtin_amdgcn_ballot_w64(cv < s) != 0ull);
      }
      asm volatile("" ::: "memory");
    };
    __builtin_amdgcn_s_setprio(LPRIO);
    while (sx + sw < 16u) {
      top();
      bool idle = true;
      const bool px = stx == S_FLY && sx <= sw;
      if (px || stw == S_ANN) {
        if (stw == S_ANN) {
          fetch(b, h2, h3, dw + sw * SLICE_BYTES);
          stw = S_FLY;
        }
        poll();
        if (px) {
          if (__builtin_expect(zq <= ZQ, 0)) clear(false);
          const unsigned ld = landed(sx, h0, h1, e2);
          room(sx);
          *(u4*)(slot + 16384) = ae;
          unsigned mn = put(a[0], h0, 0, min4(ae, 0xFFFFFFFFu));
          mn = put(a[1], h1, 4096, mn);
          asm volatile("" ::: "memory");
          stx = S_ANN;
          if (__builtin_amdgcn_ballot_w64(mn == 0u) == 0ull) {
            asm volatile("ds_write2_b32 %0, %1, %2 offset1:2" :: "v"(lds_base + L_LANDED + k * 16u), "v"(ld), "v"(e2) : "memory");
            ++sx; stx = sx == 8u ? S_END : S_X;
          } else ++retries;
        }
        top();
        idle = false;
        fnew = 0u;
      }
      const bool pw = stw == S_FLY && sw <= sx;
      if (pw || stx == S_ANN) {
        if (stx == S_ANN) {
          fetch(a, h0, h1, dx + sx * SLICE_BYTES);
          if (hasx) ae = __builtin_amdgcn_raw_buffer_load_b128(rs_my, lane * 16 + 128, (int)(de + sx * SLICE_BYTES + (ti & 3u) * 1024u), AUX_SYS);
          stx = S_FLY;
        }
        poll();
        if (pw) {
          if (__builtin_expect(zq <= ZQ, 0)) clear(false);
          const unsigned ld = landed(sw, h2, h3, EBIAS);
          room(sw);
          unsigned mn = put(b[0], h2, 8192, 0xFFFFFFFFu);
          mn = put(b[1], h3, 12288, mn);
          asm volatile("" ::: "memory");
          stw = S_ANN;
          if (__builtin_amdgcn_ballot_w64(mn == 0u) == 0ull) {
            lds_st(smem, L_LANDED + (int)k * 16 + 4, ld);
            ++sw; stw = sw == 8u ? S_END : S_W;
          } else ++retries;
        }
        idle = false;
      }
      if (idle) {
        if (zq < ZQ && sx + sw == 0u && spins >= (unsigned)ZSTART) clear(false);
        __builtin_amdgcn_s_sleep(POLL_SLEEP);
        poll();
      }
      if ((++spins & 63u) == 0u) {
        const unsigned tm = __builtin_amdgcn_readfirstlane(__builtin_amdgcn_raw_buffer_load_b32(rs_my, (int)TMO_OFF, 0, AUX_SYS));
        if (tm == call || lds_ld(smem, L_MISC) != 0u || now() - t0 > DEADLINE) { dead = true; sx = sw = 8u; }
      }
    }
    if (zq <= ZQ && !dead) clear(true);
    if (!(mode & 2u)) return;
    if (dead) {
      __builtin_amdgcn_raw_buffer_store_b32(call, rs_my, lane0, (int)TMO_OFF, AUX_SYS);
      lds_st(smem, L_MISC, 1u);
    }
    STAMP(8 + k);
    STAMPV(16 + k, retries);
    STAMPV(24 + k, tseen);
    if (k != 0u) return;
    odrop = OOR;
#pragma unroll
    for (int a = 0; a < 4; ++a)
#pragma unroll
      for (int b = 0; b < 4; ++b) acc[a][b] = f4{0.f, 0.f, 0.f, 0.f};
    acx[0] = acx[1] = f4{0.f, 0.f, 0.f, 0.f};
  } else {
  if (wk == 0) STAMPV(4, ts00);
  if (wk == 0) STAMP(2);
  if (!(mode & 2u)) return;
#pragma unroll
  for (int a = 0; a < 4; ++a)
#pragma unroll
    for (int b = 0; b < 4; ++b) acc[a][b] = f4{0.f, 0.f, 0.f, 0.f};
  acx[0] = acx[1] = f4{0.f, 0.f, 0.f, 0.f};

  unsigned done = 0;
  unsigned nd = 0;
  const char* const abase = smem + wr * 4096 + lane * 16;
  const char* const bbase = smem + 8192 + wc * 4096 + lane * 16;
  const char* const ebase = smem + 16384 + lane * 16;
  const int cons = L_CONS + (lane & 7) * 16 + wk * 4;
  const unsigned long long t0 = now();
  struct Frag { h8 b[4], a[2], e; };
  h8 af[2];
  Frag f0, f1;
  unsigned lx, lw;
  auto peek = [&]() INL {
    asm volatile("" ::: "memory");
    const unsigned long long l = lds_ld64(smem, L_LANDED + (lane & 7) * 16);
    lx = (unsigned)l; lw = (unsigned)(l >> 32);
    asm volatile("" ::: "memory");
  };
  auto take = [&](int k) INL {
    asm volatile("" ::: "memory");
    done += (lane & 7) == k ? 1u : 0u;
    lds_st(smem, cons, done);
    asm volatile("" ::: "memory");
  };
  auto read = [&](Frag& fn, int k) INL {
    asm volatile("" ::: "memory");
#pragma unroll
    for (int i = 0; i < 4; ++i) fn.b[i] = *(const h8*)(bbase + k * L_SLOT + i * 1024);
#pragma unroll
    for (int i = 0; i < 2; ++i) af[i] = *(const h8*)(abase + k * L_SLOT + i * 1024);
#pragma unroll
    for (int i = 0; i < 2; ++i) fn.a[i] = *(const h8*)(abase + k * L_SLOT + 2048 + i * 1024);
    fn.e = *(const h8*)(ebase + k * L_SLOT);
    asm volatile("" ::: "memory");
  };
  enum { W_GO = 0, W_END, W_SLOW, W_DEAD };
  auto wait_and_read = [&](Frag& fn) INL -> int {
    if (nd == 64u) return W_END;
    unsigned rdy, spins = 0;
    for (;;) {
      peek();
      rdy = (unsigned)__builtin_amdgcn_ballot_w64(min(lx & 255u, lw & 255u) > done) & 255u;
      if (rdy) break;
      if (__builtin_amdgcn_ballot_w64(min(lx, lw) >> 28 > done) != 0ull) return W_SLOW;
      if ((++spins & 63u) == 0u && (lds_ld(smem, L_MISC) != 0u || now() - t0 > 2 * DEADLINE)) return W_DEAD;
      __builtin_amdgcn_s_sleep(WSLEEP);
    }
    const int k = __builtin_ctz(rdy);
    read(fn, k);
    take(k);
    peek();
    ++nd;
    return W_GO;
  };
#define SB() __builtin_amdgcn_sched_barrier(0)
#define MF(A, B) acc[A][B] = __builtin_amdgcn_mfma_f32_16x16x32_f16((A) < 2 ? af[(A) & 1] : f.a[(A) & 1], f.b[B], acc[A][B], 0, 0, 0); SB()
  auto body = [&](auto xs, Frag& f, Frag& fn) INL -> int {
    constexpr int XS = decltype(xs)::value;
    SB();
    MF(0, 0); MF(0, 1);
    const unsigned rdy = (unsigned)__builtin_amdgcn_ballot_w64(min(lx & 255u, lw & 255u) > done) | 256u;
    SB();
    MF(0, 2);
    const int k = __builtin_ctz(rdy);
    const int ko = (k & 7) * L_SLOT;
    const char* const bp = bbase + ko;
    const char* const ap = abase + ko;
    SB();
    MF(0, 3);
    asm volatile("" ::: "memory");
    fn.b[0] = *(const h8*)(bp); SB();
    MF(1, 0);
    fn.b[1] = *(const h8*)(bp + 1024); SB();
    MF(1, 1);
    fn.b[2] = *(const h8*)(bp + 2048); SB();
    MF(1, 2);
    fn.b[3] = *(const h8*)(bp + 3072); SB();
    MF(1, 3);
    af[0] = *(const h8*)(ap); SB();
    MF(2, 0);
    af[1] = *(const h8*)(ap + 1024); SB();
    MF(2, 1);
    fn.a[0] = *(const h8*)(ap + 2048); SB();
    MF(2, 2);
    fn.a[1] = *(const h8*)(ap + 3072); SB();
    MF(2, 3);
    if (XS) fn.e = *(const h8*)(ebase + ko);
    SB();
    MF(3, 0);
    take(k); SB();
    MF(3, 1);
    peek(); SB();
    MF(3, 2); MF(3, 3);
    if (XS) {
      acx[0] = __builtin_amdgcn_mfma_f32_16x16x32_f16(f.e, f.b[XS == 1 ? 0 : 2], acx[0], 0, 0, 0);
      acx[1] = __builtin_amdgcn_mfma_f32_16x16x32_f16(f.e, f.b[XS == 1 ? 1 : 3], acx[1], 0, 0, 0);
      SB();
    }
    if (__builtin_expect(k == 8, 0)) return wait_and_read(fn);
    ++nd;
    return W_GO;
  };
  auto fast = [&](auto xs) INL -> int {
    int st = wait_and_read(f0);
    while (st == W_GO) {
      st = body(xs, f0, f1);
      if (st != W_GO) break;
      st = body(xs, f1, f0);
    }
    return st;
  };
  int st = !hasx ? fast(IC<0>{}) : wr == 0 ? fast(IC<1>{}) : fast(IC<2>{});
  for (; st == W_SLOW && nd < 64u; ++nd) {
    unsigned rdy, spins = 0;
    for (;;) {
      peek();
      rdy = (unsigned)__builtin_amdgcn_ballot_w64(min(lx, lw) >> 28 > done) & 255u;
      if (rdy) break;
      if ((++spins & 63u) == 0u && (lds_ld(smem, L_MISC) != 0u || now() - t0 > 2 * DEADLINE)) { st = W_DEAD; rdy = 1u; break; }
      __builtin_amdgcn_s_sleep(1);
    }
    const int k = __builtin_ctz(rdy);
    const unsigned se = __builtin_amdgcn_readfirstlane(lds_ld(smem, L_LANDED + k * 16 + 8));
    read(f0, k);
    take(k);
    const unsigned sx = __builtin_amdgcn_readlane(lx, k), sw = __builtin_amdgcn_readlane(lw, k);
    const int ew = (int)((sw >> (8 + 10 * wc)) & 1023u) - 2 * (int)EBIAS;
    const int e = (int)((sx >> (8 + 10 * wr)) & 1023u) + ew;
    const int e2 = (int)se + ew;
#pragma unroll
    for (int a = 0; a < 4; ++a)
#pragma unroll
      for (int b = 0; b < 4; ++b) {
        const f4 t = __builtin_amdgcn_mfma_f32_16x16x32_f16(a < 2 ? af[a & 1] : f0.a[a & 1], f0.b[b], f4{0.f, 0.f, 0.f, 0.f}, 0, 0, 0);
#pragma unroll
        for (int r = 0; r < 4; ++r) acc[a][b][r] += __builtin_ldexpf(t[r], e);
        SB();
      }
    const f4 u0 = __builtin_amdgcn_mfma_f32_16x16x32_f16(f0.e, wr ? f0.b[2] : f0.b[0], f4{0.f, 0.f, 0.f, 0.f}, 0, 0, 0);
    const f4 u1 = __builtin_amdgcn_mfma_f32_16x16x32_f16(f0.e, wr ? f0.b[3] : f0.b[1], f4{0.f, 0.f, 0.f, 0.f}, 0, 0, 0);
#pragma unroll
    for (int r = 0; r < 4; ++r) { acx[0][r] += __builtin_ldexpf(u0[r], e2); acx[1][r] += __builtin_ldexpf(u1[r], e2); }
  }
  const bool dead = st == W_DEAD;
  if (wk == 0) STAMP(5);
  if (wk == 0) STAMPV(0, __builtin_amdgcn_s_memtime() - cy00);
  if (dead) {
    const float qn = __builtin_nanf("");
#pragma unroll
    for (int a = 0; a < 4; ++a)
#pragma unroll
      for (int b = 0; b < 4; ++b) acc[a][b] = f4{qn, qn, qn, qn};
    acx[0] = acx[1] = f4{qn, qn, qn, qn};
  }
  }

  const __amdgpu_buffer_rsrc_t rs_o = __builtin_amdgcn_make_buffer_rsrc((void*)outp, 0, MM * NN * 4, RSRC3);
  const float K1 = -2.0f * 0.7978845608028654f * 1.4426950408889634f;
  const float K2 = K1 * 0.044715f;
  auto gelu2 = [&](f2 v) -> f2 {
    const f2 t = v * __builtin_elementwise_fma(v * v, f2{K2, K2}, f2{K1, K1});
    const f2 d = f2{__builtin_amdgcn_exp2f(t[0]), __builtin_amdgcn_exp2f(t[1])} + f2{1.f, 1.f};
    return v * f2{__builtin_amdgcn_rcpf(d[0]), __builtin_amdgcn_rcpf(d[1])};
  };
  const int vo = (((lane >> 4) * 4) * (NN * 4) + (lane & 15) * 16) | odrop;
  const int so = (int)((ti * 128u + (unsigned)wr * 64u) * (NN * 4) + (tj * 128u + (unsigned)wc * 64u) * 4u);
#pragma unroll
  for (int a = 0; a < 4; ++a) {
    f2 y[4][2];
#pragma unroll
    for (int b = 0; b < 4; ++b) {
      y[b][0] = gelu2(f2{acc[a][b][0], acc[a][b][1]});
      y[b][1] = gelu2(f2{acc[a][b][2], acc[a][b][3]});
    }
#pragma unroll
    for (int r = 0; r < 4; ++r) {
      const f4 o = f4{y[0][r >> 1][r & 1], y[1][r >> 1][r & 1], y[2][r >> 1][r & 1], y[3][r >> 1][r & 1]};
      st128(__builtin_bit_cast(u4, o), rs_o, vo + so + (a * 16 + r) * (NN * 4));
    }
  }
  if (hasx) {
    const int sx = (int)((15u * 128u + ti * 16u) * (NN * 4) + (tj * 128u + (unsigned)wc * 64u) * 4u) + wr * 8;
    f2 y[2][2];
#pragma unroll
    for (int b = 0; b < 2; ++b) {
      y[b][0] = gelu2(f2{acx[b][0], acx[b][1]});
      y[b][1] = gelu2(f2{acx[b][2], acx[b][3]});
    }
#pragma unroll
    for (int r = 0; r < 4; ++r) {
      const f2 o = f2{y[0][r >> 1][r & 1], y[1][r >> 1][r & 1]};
      __builtin_amdgcn_raw_buffer_store_b64(__builtin_bit_cast(u2, o), rs_o, vo + sx + r * (NN * 4), 0, AUX_SYS);
    }
  }
}

}

extern "C" void kernel_launch(int rank, int nranks, ncclComm_t comm,
                              hipStream_t stream,
                              void* const* d_in, const int* in_sizes, int n_in,
                              void* d_out, int out_size,
                              const int* in_dtypes, int out_dtype,
                              void* const* d_out_all,
                              void* const* d_ws_all, size_t ws_bytes) {
  (void)comm; (void)in_dtypes; (void)out_dtype; (void)d_out_all;
  if (nranks != NR || (unsigned)rank >= (unsigned)NR || n_in < 2 || ws_bytes < (size_t)WS_NEED ||
      in_sizes[0] != MM * KS || in_sizes[1] != KS * NN || out_size != MM * NN) {
    fprintf(stderr, "kernel_launch: unexpected problem shape\n");
    return;
  }
  static int ncu = 0;
  if (ncu == 0) {
    int dev = 0, n = 0;
    HIP_CHECK(hipGetDevice(&dev));
    HIP_CHECK(hipDeviceGetAttribute(&n, hipDeviceAttributeMultiprocessorCount, dev));
    ncu = n > 0 ? n : 1;
  }
  Params p;
  p.x = (const float*)d_in[0];
  p.w = (const float*)d_in[1];
  p.out = (float*)d_out;
  for (int r = 0; r < NR; ++r) p.ws[r] = (char*)d_ws_all[(rank + r) % NR];
  p.my = p.ws[0];
  p.rank = rank;
  if (ncu >= NWG) {
    p.mode = 3;
    hipLaunchKernelGGL(fused_gemm_ag, dim3(NWG), dim3(NWAVE * 64), 0, stream, p);
  } else {
    p.mode = 1;
    hipLaunchKernelGGL(fused_gemm_ag, dim3(NWG), dim3(NWAVE * 64), 0, stream, p);
    p.mode = 2;
    hipLaunchKernelGGL(fused_gemm_ag, dim3(NWG), dim3(NWAVE * 64), 0, stream, p);
  }
  HIP_CHECK(hipGetLastError());
}
```
